```python
import numpy as np

kernel_name = "gpu_a2a_gemm_m4096_k4096_n4096_bf16_reluq8"


ABSMAX_RTOL = 0.02

COMPUTE_DTYPE = "bfloat16"
EPILOGUE = "relu_quant_int8"
IN_QUANT = None
OUT_QUANT = "int8"
IN_DTYPES = {"x": "bfloat16", "w_mat": "bfloat16"}
WORLD_SIZE = 8

def _to_bf16(x):
    u = np.ascontiguousarray(x, dtype=np.float32).view(np.uint32)
    return ((u + 0x7FFF + ((u >> 16) & 1)) & 0xFFFF0000).view(np.float32)

def setup_inputs():

    x = _to_bf16(np.random.randn(4096, 4096).astype(np.float32))
    w_mat = _to_bf16((np.random.randn(4096, 4096) / np.sqrt(4096)).astype(np.float32))
    return {"x": x, "w_mat": w_mat}

def _quant_dequant(y, amax):
    scale = amax / 127.0
    q = np.clip(np.round(y / scale), -127.0, 127.0)
    return (q * scale).astype(np.float32)
def _epilogue(y):
    y = np.maximum(y, 0.0)
    return _quant_dequant(y, np.abs(y).max())

def reference(x, w_mat):
    y = (x @ w_mat).astype(np.float32)
    return _epilogue(y)
```

```cpp
#include <hip/hip_runtime.h>
#include <rccl/rccl.h>
#include <cstdio>

#define HIP_CHECK(expr) do { hipError_t _e = (expr); if (_e != hipSuccess) { \
    fprintf(stderr, "HIP error %s at %s:%d\n", hipGetErrorString(_e), __FILE__, __LINE__); \
    return; } } while (0)

typedef unsigned u32x4 __attribute__((ext_vector_type(4)));
typedef unsigned u32x2 __attribute__((ext_vector_type(2)));
typedef short bf16x8 __attribute__((ext_vector_type(8)));
typedef short s16x4 __attribute__((ext_vector_type(4)));
typedef float f32x4 __attribute__((ext_vector_type(4)));
#define LAS __attribute__((address_space(3)))
#define GAS __attribute__((address_space(1)))

constexpr int NR = 8;
constexpr int NWG = 256;
constexpr size_t WS_CNT = 0;
constexpr size_t WS_TMO = 32768;
constexpr size_t WS_FLAG = 36864;
constexpr size_t WS_AMAX = 65536;
constexpr size_t WS_A = 1 << 20;
constexpr size_t WS_END = WS_A + (4 << 20);
constexpr unsigned long long DEADLINE = 50000000ull;

struct Params {
  const char* x;
  const char* w;
  char* out;
  char* ws[NR];
  int rank;
  int pad;
};

#define GLDS(src, dst, aux) __builtin_amdgcn_global_load_lds((const GAS unsigned*)(src), (LAS unsigned*)(dst), 16, 0, (aux))
#define WAITV(n) asm volatile("s_waitcnt vmcnt(" #n ")" ::: "memory")
#define WAITL(n) asm volatile("s_waitcnt lgkmcnt(" #n ")" ::: "memory")

__device__ __forceinline__ void st16_sys(void* p, u32x4 v) {
  asm volatile("global_store_dwordx4 %0, %1, off sc0 sc1\n\ts_nop 1" ::"v"(p), "v"(v) : "memory");
}

constexpr int SLOT = 24576;
constexpr int NSLOT = 4;
constexpr int LDS_CTRL = SLOT * NSLOT;
constexpr int LDS_BYTES = LDS_CTRL + 256;

__global__ void __launch_bounds__(256) a2a_gemm(Params p) {
  extern __shared__ __attribute__((aligned(16))) char lds_[];
  LAS char* lds = (LAS char*)lds_;
  const int tid = threadIdx.x, wave = __builtin_amdgcn_readfirstlane(tid >> 6), lane = tid & 63;
  const int b = blockIdx.x, xcd = b & 7, idx = b >> 3, mt = idx & 7, nt = idx >> 3;
  const int rank = p.rank;
  char* ws = p.ws[rank];
  unsigned* tmo = (unsigned*)(ws + WS_TMO);
  unsigned* cntp = (unsigned*)(ws + WS_CNT + b * 128);
  const unsigned call = __hip_atomic_load(cntp, __ATOMIC_RELAXED, __HIP_MEMORY_SCOPE_SYSTEM) + 1;

  {
    const int q = b & 7, rb = b >> 3;
    const char* src = p.x + (size_t)(q * 512 + rb * 16 + (lane & 15)) * 1024 + (lane >> 4) * 16;
    char* dst = p.ws[q] + WS_A + (size_t)rank * (512 << 10) + rb * 1024 + lane * 16;
    u32x4 v[4];
#pragma unroll
    for (int j = 0; j < 4; ++j) v[j] = *(const GAS u32x4*)(src + (wave * 4 + j) * 64);
#pragma unroll
    for (int j = 0; j < 4; ++j) st16_sys(dst + (size_t)(wave * 4 + j) * 32768, v[j]);
    WAITV(0);
    __syncthreads();
    if (tid == 0) {
      __hip_atomic_store((unsigned*)(p.ws[q] + WS_FLAG) + rb * 8 + rank, call, __ATOMIC_RELAXED, __HIP_MEMORY_SCOPE_SYSTEM);
      __hip_atomic_store(cntp, call, __ATOMIC_RELAXED, __HIP_MEMORY_SCOPE_SYSTEM);
    }
  }
  if (wave == 0) {
    unsigned* f = (unsigned*)(ws + WS_FLAG) + mt * 32 + (lane & 31);
    unsigned long long t0 = __builtin_amdgcn_s_memrealtime();
    while (true) {
      unsigned v = __hip_atomic_load(f, __ATOMIC_RELAXED, __HIP_MEMORY_SCOPE_SYSTEM);
      if (__all(v >= call)) break;
      if (__hip_atomic_load(tmo, __ATOMIC_RELAXED, __HIP_MEMORY_SCOPE_AGENT) == call ||
          __builtin_amdgcn_s_memrealtime() - t0 > DEADLINE) {
        __hip_atomic_store(tmo, call, __ATOMIC_RELAXED, __HIP_MEMORY_SCOPE_AGENT);
        break;
      }
      __builtin_amdgcn_s_sleep(2);
    }
  }
  __syncthreads();

  const int wm = wave & 1, wn = wave >> 1;
  const char* abase = ws + WS_A + mt * 4096 + lane * 16;
  const int ncol0 = xcd * 512 + nt * 128;
  auto stage = [&](int s) {
    LAS char* sl = lds + (s % NSLOT) * SLOT;
#pragma unroll
    for (int j = 0; j < 2; ++j) {
      const int pc = wave * 2 + j;
      GLDS(abase + (size_t)(s * 2 + (pc >> 2)) * 32768 + (pc & 3) * 1024, sl + pc * 1024, 17);
    }
#pragma unroll
    for (int j = 0; j < 4; ++j) {
      const int pc = wave * 4 + j;
      const int row = pc * 4 + (lane >> 4);
      const int ch = (lane & 15) ^ (((row & 3) << 2) | ((row >> 2) & 3));
      GLDS(p.w + (size_t)(s * 64 + row) * 8192 + ncol0 * 2 + ch * 16, sl + 8192 + pc * 1024, 0);
    }
  };
  f32x4 acc[2][4];
#pragma unroll
  for (int i = 0; i < 2; ++i)
#pragma unroll
    for (int j = 0; j < 4; ++j) acc[i][j] = f32x4{0.f, 0.f, 0.f, 0.f};
  stage(0); stage(1); stage(2);
  const int g = lane >> 4, tq = (lane & 15) >> 2, tp = lane & 3;
  for (int s = 0; s < 64; ++s) {
    if (s < 62) WAITV(12); else if (s == 62) WAITV(6); else WAITV(0);
    __builtin_amdgcn_s_barrier();
    if (s + 3 < 64) stage(s + 3);
    LAS char* sl = lds + (s % NSLOT) * SLOT;
#pragma unroll
    for (int ks = 0; ks < 2; ++ks) {
      bf16x8 xa[2], wb[4];
#pragma unroll
      for (int i = 0; i < 2; ++i) xa[i] = *(LAS bf16x8*)(sl + (ks * 4 + wm * 2 + i) * 1024 + lane * 16);
#pragma unroll
      for (int j = 0; j < 4; ++j) {
        s16x4 h[2];
#pragma unroll
        for (int t = 0; t < 2; ++t) {
          const int row = 32 * ks + 8 * g + 4 * t + tq;
          const int ch = 2 * (wn * 4 + j) + (tp >> 1);
          const int off = 256 * row + 16 * (ch ^ (((row & 3) << 2) | ((row >> 2) & 3))) + 8 * (tp & 1);
          h[t] = __builtin_amdgcn_ds_read_tr16_b64_v4i16((LAS s16x4*)(sl + 8192 + off));
        }
        wb[j] = bf16x8{h[0][0], h[0][1], h[0][2], h[0][3], h[1][0], h[1][1], h[1][2], h[1][3]};
      }
#pragma unroll
      for (int i = 0; i < 2; ++i)
#pragma unroll
        for (int j = 0; j < 4; ++j)
          acc[i][j] = __builtin_amdgcn_mfma_f32_16x16x32_bf16(wb[j], xa[i], acc[i][j], 0, 0, 0);
    }
    WAITL(0);
  }
  float m = 0.f;
#pragma unroll
  for (int i = 0; i < 2; ++i)
#pragma unroll
    for (int j = 0; j < 4; ++j)
#pragma unroll
      for (int e = 0; e < 4; ++e) m = fmaxf(m, acc[i][j][e]);
#pragma unroll
  for (int o = 32; o >= 1; o >>= 1) m = fmaxf(m, __shfl_xor(m, o));
  volatile LAS float* red = (volatile LAS float*)(lds + LDS_CTRL);
  if (lane == 0) red[wave] = m;
  __syncthreads();
  if (tid < NR) {
    float mm = fmaxf(fmaxf(red[0], red[1]), fmaxf(red[2], red[3]));
    unsigned long long gr = ((unsigned long long)call << 32) | __float_as_uint(mm);
    __hip_atomic_store((unsigned long long*)(p.ws[tid] + WS_AMAX) + rank * NWG + b, gr, __ATOMIC_RELAXED, __HIP_MEMORY_SCOPE_SYSTEM);
  }
  {
    unsigned long long* gp = (unsigned long long*)(ws + WS_AMAX) + tid;
    unsigned long long t0 = __builtin_amdgcn_s_memrealtime();
    unsigned mx;
    while (true) {
      bool ok = true;
      mx = 0;
#pragma unroll
      for (int r = 0; r < NR; ++r) {
        unsigned long long v = __hip_atomic_load(gp + r * NWG, __ATOMIC_RELAXED, __HIP_MEMORY_SCOPE_SYSTEM);
        ok &= (unsigned)(v >> 32) >= call;
        mx = max(mx, (unsigned)v);
      }
      if (__all(ok)) break;
      if (__hip_atomic_load(tmo, __ATOMIC_RELAXED, __HIP_MEMORY_SCOPE_AGENT) == call ||
          __builtin_amdgcn_s_memrealtime() - t0 > DEADLINE) {
        __hip_atomic_store(tmo, call, __ATOMIC_RELAXED, __HIP_MEMORY_SCOPE_AGENT);
        break;
      }
      __builtin_amdgcn_s_sleep(1);
    }
#pragma unroll
    for (int o = 32; o >= 1; o >>= 1) mx = max(mx, (unsigned)__shfl_xor((int)mx, o));
    if (lane == 0) red[4 + wave] = __uint_as_float(mx);
  }
  __syncthreads();
  const float amax = fmaxf(fmaxf(red[4], red[5]), fmaxf(red[6], red[7]));
  const bool bad = __hip_atomic_load(tmo, __ATOMIC_RELAXED, __HIP_MEMORY_SCOPE_AGENT) == call;
  const float inv = amax > 0.f ? 127.0f / amax : 0.f;
#pragma unroll
  for (int i = 0; i < 2; ++i) {
    char* orow = p.out + (size_t)(mt * 64 + wm * 32 + i * 16 + (lane & 15)) * 4096 + ncol0 + wn * 64 + g * 4;
#pragma unroll
    for (int j = 0; j < 4; ++j) {
      unsigned pk = 0;
#pragma unroll
      for (int e = 0; e < 4; ++e) {
        float qf = fminf(fmaxf(__builtin_rintf(acc[i][j][e] * inv), 0.f), 127.f);
        pk |= ((unsigned)qf) << (8 * e);
      }
      if (bad) pk = 0x80808080u;
      *(GAS unsigned*)(orow + j * 16) = pk;
    }
  }
}

extern "C" void kernel_launch(int rank, int nranks, ncclComm_t comm, hipStream_t stream, void* const* d_in,
                              const int* in_sizes, int n_in, void* d_out, int out_size, const int* in_dtypes,
                              int out_dtype, void* const* d_out_all, void* const* d_ws_all, size_t ws_bytes) {
  if (nranks != NR || ws_bytes < WS_END || (unsigned)rank >= (unsigned)NR) return;
  static bool init = false;
  if (!init) {
    HIP_CHECK(hipFuncSetAttribute((const void*)a2a_gemm, hipFuncAttributeMaxDynamicSharedMemorySize, LDS_BYTES));
    init = true;
  }
  Params p;
  p.x = (const char*)d_in[0];
  p.w = (const char*)d_in[1];
  p.out = (char*)d_out;
  for (int i = 0; i < NR; ++i) p.ws[i] = (char*)d_ws_all[i];
  p.rank = rank;
  p.pad = 0;
  hipLaunchKernelGGL(a2a_gemm, dim3(NWG), dim3(256), LDS_BYTES, stream, p);
  HIP_CHECK(hipGetLastError());
}
```

```cpp
#include <hip/hip_runtime.h>
#include <rccl/rccl.h>
#include <cstdio>
#include <cstddef>

#define HIP_CHECK(expr) do { hipError_t _e = (expr); if (_e != hipSuccess) { \
    fprintf(stderr, "HIP error %s at %s:%d\n", hipGetErrorString(_e), __FILE__, __LINE__); \
    return; } } while (0)

typedef unsigned u32x4 __attribute__((ext_vector_type(4)));
typedef unsigned u32x2 __attribute__((ext_vector_type(2)));
typedef float f32x16 __attribute__((ext_vector_type(16)));
typedef unsigned long long u64;
#define LAS __attribute__((address_space(3)))
#define GAS __attribute__((address_space(1)))

#define PUSH_DELTA 33
#define PUSH_T0 90
#define PUSH_HOLD 4
#define PUSH_HOLDX 60
#define PSLEEP 4
#define GUESS_MIN 8
#define WT_MAX 56
#define XWIN 16
#define WWIN 24

constexpr int NR = 8, NWG = 256, NTHR = 512;
constexpr unsigned WS_CNT = 0;
constexpr unsigned WS_AM1 = 65536;
constexpr unsigned WS_AM2 = 98304;
constexpr unsigned WS_TMO = 114688;
constexpr unsigned WS_XF = 131072;
constexpr unsigned BLK = 16384 + 128;
constexpr unsigned WS_A = 1 << 20;
constexpr unsigned WS_XD = 6 << 20;
constexpr size_t WS_END = 18 << 20;
static_assert(WS_A + 256 * BLK <= WS_XD && WS_XD + 256 * 49152 <= WS_END && WS_XF + 256 * 2048 <= WS_A, "layout");
constexpr unsigned DEADLINE = 40000000u;
#define SPINCAP "0x1000000"

constexpr int LDS_W = 0xc000, LDS_CTRL = 0x24000;
constexpr int C_DONE = LDS_CTRL;
constexpr int C_FULL = LDS_CTRL + 1024;
constexpr int C_BAD = LDS_CTRL + 1536;
constexpr int C_OUT = C_BAD + 4;
constexpr int C_GFLAG = C_BAD + 8;
constexpr int C_JUNK = LDS_CTRL + 1792;
constexpr int LDS_BYTES = C_JUNK + 256;
constexpr unsigned POISON = 0xFFFFFFFFu;

struct Params {
  const char* x;
  const char* w;
  char* out;
  u64* dbg;
  int rank;
  int pad0, pad1, pad2;
  char* wsrel[NR];
  u64 pad3, pad4;
  u64 sched[16][2];
};

__host__ __device__ constexpr int seq_pos(int j, int odd) { return !odd || j < 6 ? j : j < 14 ? j + 2 : j - 8; }
__host__ __device__ constexpr int local_g(int s, int mt, int sq, int nt) { return 4 * ((mt + nt + s) & 1) + ((0x3120 >> (4 * sq)) & 3); }
static void make_sched(u64 (*t)[2]) {
  for (int c = 0; c < 16; ++c) {
    const int mt = c & 3, sq = c >> 2;
    int o = 0;
    t[c][0] = t[c][1] = 0;
    auto put = [&](int dg) { t[c][o >> 3] |= (u64)dg << (8 * (o & 7)); ++o; };
    for (int i = 0; i < 8; ++i) {
      const int g = i >> 2, pp = (mt - i) & 3, d = 2 * pp + (((sq >> 1) + g) & 1);
      if (d && (pp & 1) == (sq & 1)) put(d * 8 + g);
    }
    for (int j = 4; j < 16; ++j) {
      const int jj = seq_pos(j, sq >> 1), G = jj >> 2, d = 2 * ((mt - jj) & 3) + ((sq + G) & 1);
      if (d) put(d * 8 + 2 * G + (sq >> 1));
    }
  }
}

#define STR_(x) #x
#define STR(x) STR_(x)
#define RT() __builtin_amdgcn_s_memrealtime()

__device__ __forceinline__ unsigned uni(unsigned v) { return __builtin_amdgcn_readfirstlane(v); }
__device__ __forceinline__ unsigned wrlane(unsigned val, int l, unsigned old) {
  asm("v_writelane_b32 %0, %1, %2" : "+v"(old) : "s"(val), "n"(l));
  return old;
}
__device__ __forceinline__ u64 ld64_sys(const void* ptr) {
  return __hip_atomic_load((const GAS u64*)ptr, __ATOMIC_RELAXED, __HIP_MEMORY_SCOPE_SYSTEM);
}
__device__ __forceinline__ void st_sys(void* ptr, unsigned v) {
  __hip_atomic_store((GAS unsigned*)ptr, v, __ATOMIC_RELAXED, __HIP_MEMORY_SCOPE_SYSTEM);
}
__device__ __forceinline__ void st64_sys(void* ptr, u64 v) {
  __hip_atomic_store((GAS u64*)ptr, v, __ATOMIC_RELAXED, __HIP_MEMORY_SCOPE_SYSTEM);
}
__device__ __forceinline__ unsigned ld_tmo(char* ws) {
  return __hip_atomic_load((GAS unsigned*)(ws + WS_TMO), __ATOMIC_RELAXED, __HIP_MEMORY_SCOPE_AGENT);
}
__device__ __forceinline__ void st_tmo(char* ws, unsigned call) {
  __hip_atomic_store((GAS unsigned*)(ws + WS_TMO), call, __ATOMIC_RELAXED, __HIP_MEMORY_SCOPE_AGENT);
}
__device__ __forceinline__ bool expired(char* ws, unsigned call, unsigned t0) {
  if (uni(ld_tmo(ws)) == call || (unsigned)RT() - t0 > DEADLINE) {
    st_tmo(ws, call);
    return true;
  }
  return false;
}
__device__ __forceinline__ unsigned wave_umax(unsigned v) {
  v = max(v, (unsigned)__builtin_amdgcn_update_dpp(0, (int)v, 0x111, 0xf, 0xf, true));
  v = max(v, (unsigned)__builtin_amdgcn_update_dpp(0, (int)v, 0x112, 0xf, 0xf, true));
  v = max(v, (unsigned)__builtin_amdgcn_update_dpp(0, (int)v, 0x114, 0xf, 0xf, true));
  v = max(v, (unsigned)__builtin_amdgcn_update_dpp(0, (int)v, 0x118, 0xf, 0xf, true));
  const unsigned a = __builtin_amdgcn_readlane(v, 15), c = __builtin_amdgcn_readlane(v, 31);
  const unsigned d = __builtin_amdgcn_readlane(v, 47), e = __builtin_amdgcn_readlane(v, 63);
  return max(max(a, c), max(d, e));
}

#define OUT_STORE(ptr, val, sc) asm volatile("global_store_dwordx4 %0, %1, off " sc "\n\ts_nop 1" ::"v"(ptr), "v"(val) : "memory")

#define NL "\n\t"
#define LOAD_CALL "s_load_dword %[call], s[70:71], %[cnt] glc" NL
#define ARGS_IN "s_waitcnt lgkmcnt(0)" NL
#define SET_BAD(va, vb) "v_mov_b32 " va ", %[cbad]" NL "v_mov_b32 " vb ", -1" NL "ds_write_b32 " va ", " vb NL "s_waitcnt lgkmcnt(0)" NL
#define WAIT_TIME(l, t) l "%=:" NL "s_sleep 1" NL "s_memrealtime s[96:97]" NL "s_waitcnt lgkmcnt(0)" NL \
        "s_sub_u32 s96, s96, s74" NL "s_cmp_lt_u32 s96, " t NL "s_cbranch_scc1 " l "%=" NL
#define WIN "s_waitcnt vmcnt(%[win])" NL
#define M0NEXT "s_add_u32 m0, m0, 0x1000" NL WIN
#define DMA(r, o, sc) "global_load_lds_dwordx4 v" #r ", s[98:99] offset:" #o sc NL
#define DMA4(a, b, c, d, sc) DMA(a, 0, sc) DMA(b, 1024, sc) DMA(c, 2048, sc) DMA(d, 3072, sc)
#define PUBLISH "s_mov_b32 m0, %[full]" NL "s_nop 0" NL "global_load_lds_dword v38, %[tbl]" NL
#define WAIT_FREE \
        "Lfree%=:" NL "ds_read_b32 v36, %[chka]" NL "s_waitcnt lgkmcnt(0)" NL "v_cmp_lt_i32 vcc, v36, v37" NL \
        "s_cbranch_vccnz Lspin%=" NL "Lgo%=:" NL
#define SPIN \
        "Lspin%=:" NL "s_sleep 1" NL "s_add_u32 %[spin], %[spin], 1" NL "s_cmp_lt_u32 %[spin], " SPINCAP NL \
        "s_cbranch_scc1 Lfree%=" NL SET_BAD("v36", "v39") "s_branch Lgo%=" NL
#define NEXT_SLOT(lo, hi) "s_add_u32 %[ld], %[ld], 0x4000" NL "s_cmp_eq_u32 %[ld], " hi NL "s_cselect_b32 %[ld], " lo ", %[ld]" NL
#define NEXT_DG "s_lshr_b64 s[92:93], s[92:93], 8" NL "s_lshl_b32 %[off], s94, 24" NL "s_or_b32 s93, s93, %[off]" NL \
        "s_lshr_b64 s[94:95], s[94:95], 8" NL

__global__ void __launch_bounds__(NTHR) a2a_gemm(Params p) {
  extern __shared__ __attribute__((aligned(16))) char lds_[];
  LAS char* lds = (LAS char*)lds_;
  const int b = blockIdx.x, sq = b & 3, idx = b >> 3, mt = idx & 3, ntl = idx >> 2;
  const int dd = (b - idx) & 7;
  u32x4 a_xw, a_sc;
  u32x2 a_ws, a_wq, a_t0;
  unsigned a_rank;
  asm volatile("s_memrealtime %4" NL "s_load_dwordx4 %0, %6, 0x0" NL "s_load_dwordx4 %1, %6, %8 offset:0x80" NL
               "s_load_dword %5, %6, 0x20" NL "s_load_dwordx2 %2, %6, 0x30" NL "s_load_dwordx2 %3, %6, %7 offset:0x30"
               : "=&{s[64:67]}"(a_xw), "=&{s[76:79]}"(a_sc), "=&{s[70:71]}"(a_ws), "=&{s[72:73]}"(a_wq), "=&{s[74:75]}"(a_t0),
                 "=&{s68}"(a_rank)
               : "s"(__builtin_amdgcn_kernarg_segment_ptr()), "s"(dd * 8), "s"((sq * 4 + mt) * 16));
#define ARGS "{s[64:67]}"(a_xw), "{s[76:79]}"(a_sc), "{s[70:71]}"(a_ws), "{s[72:73]}"(a_wq), "{s[74:75]}"(a_t0), "{s68}"(a_rank)
  static_assert(offsetof(Params, rank) == 0x20 && offsetof(Params, wsrel) == 0x30 && offsetof(Params, sched) == 0x80, "offsets");
  if (dd == 0 && idx < 8 && threadIdx.x < 64) {
    unsigned vo = threadIdx.x * 128;
    asm volatile("s_getpc_b64 s[96:97]" NL "s_mov_b32 m0, %1" NL "s_and_b32 s96, s96, 0xffffff00" NL
                 "global_load_lds_dword %0, s[96:97]" NL "v_add_u32 %0, 0x2000, %0" NL "global_load_lds_dword %0, s[96:97]" NL
                 "v_add_u32 %0, 0x800, %0" NL "global_load_lds_dword %0, s[96:97]"
                 : "+v"(vo) : "n"(C_JUNK) : "memory", "scc", "s96", "s97");
  }
  const int tid = threadIdx.x, wave = uni(tid >> 6), lane = tid & 63;
  const int n0 = ((b & 7) >> 2) * 2048 + ntl * 256;
  alignas(256) static constexpr unsigned IOTA[64] = {
      0,  1,  2,  3,  4,  5,  6,  7,  8,  9,  10, 11, 12, 13, 14, 15, 16, 17, 18, 19, 20, 21, 22, 23, 24, 25, 26, 27, 28, 29, 30, 31,
      32, 33, 34, 35, 36, 37, 38, 39, 40, 41, 42, 43, 44, 45, 46, 47, 48, 49, 50, 51, 52, 53, 54, 55, 56, 57, 58, 59, 60, 61, 62, 63};
  const int pi = dd ? idx : 32;
  const int tdel = PUSH_T0 + pi * PUSH_DELTA;
  unsigned call = 0;

  int wv = (0x9 >> wave) & 1;
  asm volatile("" : "+s"(wv));
  if (wv) {
    const int hf = wave & 1;
    const int pj = idx < 8 ? idx : seq_pos(idx >> 1, idx & 1);
    const int pg = idx < 8 ? idx >> 2 : (pj >> 2) * 2 + (idx & 1), pmt = (pj + (dd >> 1)) & 3;
    unsigned so0 = (lane >> 3) * 1024 + (((lane & 7) ^ (lane >> 4)) << 4);
    unsigned so1 = (so0 ^ 64) + 8192, dof = lane * 16 + hf * 8192;
    unsigned to = WS_XF + (b * 16 + lane) * 128;
    to = wrlane(WS_AM1 + b * 128, 16, to);
    to = wrlane(WS_AM2 + (b & 63) * 128, 17, to);
    to = wrlane(WS_TMO, 18, to);
    to = wrlane(WS_A + b * BLK + 16384, 19, to);
#define XLD(r, s) "global_load_dwordx4 v[" #r ":" #r "+3], %[so" #s "], s[98:99] nt" NL "v_add_u32 %[so" #s "], 0x4000, %[so" #s "]" NL
#define XST(r, o) "s_waitcnt vmcnt(7)" NL "global_store_dwordx4 %[dof], v[" #r ":" #r "+3], s[90:91] offset:" #o " sc0 sc1" NL
    asm volatile(
        "s_setprio 3" NL
        ARGS_IN
        LOAD_CALL
        "s_sub_u32 s98, s68, %[dd]" NL "s_and_b32 s98, s98, 7" NL "s_lshl_b32 s98, s98, 19" NL "s_add_u32 s98, s98, %[sof]" NL
        "s_add_u32 s98, s64, s98" NL "s_addc_u32 s99, s65, 0" NL
        "s_add_u32 s90, s72, %[bof]" NL "s_addc_u32 s91, s73, 0" NL
        "s_cmp_eq_u32 %[hf], 0" NL
        "s_cbranch_scc1 Lnz%=" NL
        "v_mov_b32 v64, 0" NL
        "ds_write_b32 %[za], v64" NL
        "s_mov_b32 m0, %[junk]" NL
        "s_mov_b64 exec, 0xfffff" NL
        "global_load_lds_dword %[to], s[70:71] sc0 sc1" NL
        "s_mov_b64 exec, -1" NL
        "Lnz%=:" NL
        "s_cmp_lt_u32 %[pi], " STR(PUSH_HOLD) NL
        "s_cbranch_scc1 Lld%=" NL
        "s_waitcnt lgkmcnt(0)" NL
        "s_barrier" NL
        "s_cmp_lt_u32 %[pi], 32" NL
        "s_cbranch_scc0 Lnl%=" NL
        "Lld%=:" NL
        XLD(64, 0) XLD(68, 1) XLD(72, 0) XLD(76, 1) XLD(80, 0) XLD(84, 1) XLD(88, 0) XLD(92, 1)
        "s_cmp_lt_u32 %[pi], " STR(PUSH_HOLD) NL
        "s_cbranch_scc0 Lnl%=" NL
        "s_waitcnt lgkmcnt(0)" NL
        "s_barrier" NL
        "Lnl%=:" NL
        "s_add_u32 %[call], %[call], 1" NL
        "s_cmp_lt_u32 %[pi], 32" NL
        "s_cbranch_scc0 Lend%=" NL
        WAIT_TIME("Lw", "%[tst]")
        XST(64, 0) XST(68, 1024) XST(72, 2048) XST(76, 3072)
        "v_add_u32 %[dof], 0x1000, %[dof]" NL
        XST(80, 0) XST(84, 1024) XST(88, 2048) XST(92, 3072)
        "s_mov_b64 exec, 1" NL
        "v_mov_b32 %[so0], %[fo]" NL
        "v_mov_b32 %[so1], %[call]" NL
        "s_waitcnt vmcnt(0)" NL
        "global_store_dword %[so0], %[so1], s[90:91] sc0 sc1" NL
        "s_mov_b64 exec, -1" NL
        "Lend%=:" NL
        "s_cmp_eq_u32 %[hf], 0" NL
        "s_cbranch_scc0 Lout%=" NL
        "s_cmp_lt_u32 %[pi], 32" NL
        "s_cbranch_scc1 Lfin%=" NL
        WAIT_TIME("Lc", "%[tcw]")
        "s_getpc_b64 s[96:97]" NL
        "s_mov_b32 m0, %[junk]" NL
        "v_lshlrev_b32 v65, 3, %[dof]" NL
        "global_load_lds_dword v65, s[96:97]" NL
        "v_add_u32 v65, 0x2000, v65" NL
        "global_load_lds_dword v65, s[96:97]" NL
        "Lfin%=:" NL
        "s_endpgm" NL
        "Lout%=:"
        : [so0] "+&v"(so0), [so1] "+&v"(so1), [dof] "+&v"(dof), [call] "=&s"(call)
        : [sof] "s"((pmt * 128 + hf * 64) * 1024 + pg * 128), [bof] "s"(WS_A + ((dd * 8 + pg) * 4 + pmt) * BLK), [dd] "s"(dd),
          [cnt] "s"(WS_CNT + b * 128), [pi] "s"(pi), [tst] "s"(tdel), [tcw] "s"(idx < 16 ? 700 : 1100), [hf] "s"(hf),
          [fo] "s"(16384 + 4 * hf), [za] "v"(LDS_CTRL + (lane < 6 ? lane * 256 : 1536 + (lane & 7) * 4)), [junk] "n"(C_JUNK),
          [to] "v"(to), ARGS
        : "memory", "scc", "s90", "s91", "s92", "s93", "s96", "s97", "s98", "s99",
          "v64", "v65", "v66", "v67", "v68", "v69", "v70", "v71", "v72", "v73", "v74", "v75", "v76", "v77", "v78", "v79",
          "v80", "v81", "v82", "v83", "v84", "v85", "v86", "v87", "v88", "v89", "v90", "v91", "v92", "v93", "v94", "v95");
  }
  if (wave == 1) {
    const unsigned vl = (lane >> 3) * 1024 + (((lane & 7) ^ (lane >> 4)) << 4);
    unsigned s_step, s_ld, s_off, s_spin;
#define XLOCAL(g, slot) \
        "s_lshl_b32 s98, s68, 19" NL "s_add_u32 s98, s98, " g NL "s_add_u32 s98, s64, s98" NL "s_addc_u32 s99, s65, 0" NL "s_mov_b32 m0, " slot NL WIN \
        DMA4(40, 41, 42, 43, "") M0NEXT DMA4(44, 45, 46, 47, "") M0NEXT DMA4(48, 49, 50, 51, "") M0NEXT DMA4(52, 53, 54, 55, "")
    asm volatile(
        "v_add_u32 v40, %[r0], %[vl]" NL
        "v_xor_b32 v41, 64, v40" NL "v_add_u32 v41, 0x1c00, v41" NL
        "v_add_u32 v42, 0x3800, v40" NL "v_add_u32 v43, 0x3800, v41" NL
        ".irp r,44,45,46,47,48,49,50,51,52,53,54,55" NL
        "v_add_u32 v\\r, 0x8000, v[\\r-4]" NL "v_and_b32 v\\r, 0x1ffff, v\\r" NL
        ".endr" NL
        ARGS_IN
        LOAD_CALL
        "s_cmp_lt_u32 %[pi], " STR(PUSH_HOLD) NL
        "s_cbranch_scc0 Lnh%=" NL
        "s_barrier" NL
        WAIT_TIME("Lh", "%[th]")
        "Lnh%=:" NL
        XLOCAL("%[g0]", "0")
        "s_cmp_lt_u32 %[pi], " STR(PUSH_HOLD) NL
        "s_cbranch_scc1 Lnb%=" NL
        "s_barrier" NL
        "Lnb%=:" NL
        "v_mov_b32 v38, 8" NL PUBLISH
        XLOCAL("%[g1]", "0x4000")
        "v_mov_b32 v38, 16" NL PUBLISH
        "v_add_u32 v40, %[q0], %[l16]" NL
        ".irp r,41,42,43" NL
        "v_add_u32 v\\r, 0x1000, v[\\r-1]" NL "v_and_b32 v\\r, 0x3fff, v\\r" NL
        ".endr" NL
        "v_mov_b32 v37, 0" NL
        "s_mov_b64 s[92:93], s[76:77]" NL "s_mov_b64 s[94:95], s[78:79]" NL
        "s_waitcnt lgkmcnt(0)" NL
        "s_add_u32 %[call], %[call], 1" NL
        "s_mov_b32 %[step], 2" NL "s_mov_b32 %[ld], 0x8000" NL "s_mov_b32 %[spin], 0" NL
        "Ltop%=:" NL
        "s_and_b32 %[off], s92, 63" NL
        "s_mul_i32 %[off], %[off], %[blk4]" NL
        "s_add_u32 %[off], %[off], %[aof]" NL
        "s_add_u32 s98, s70, %[off]" NL
        "s_addc_u32 s99, s71, 0" NL
        NEXT_DG
        "s_mov_b32 m0, %[ld]" NL
        "Lpoll%=:" NL
        "s_load_dwordx2 s[90:91], s[98:99], 0x4000 glc" NL
        "s_memrealtime s[96:97]" NL
        "s_waitcnt lgkmcnt(0)" NL
        "s_min_u32 s90, s90, s91" NL
        "s_cmp_ge_u32 s90, %[call]" NL
        "s_cbranch_scc1 Lfree%=" NL
        "s_sleep " STR(PSLEEP) NL
        "s_sub_u32 s96, s96, s74" NL
        "s_cmp_lt_u32 s96, %[dl]" NL
        "s_cbranch_scc1 Lpoll%=" NL
        "v_mov_b32 v36, %[tmo]" NL "v_mov_b32 v39, %[call]" NL
        "global_store_dword v36, v39, s[70:71] sc1" NL
        SET_BAD("v36", "v39")
        WAIT_FREE
        WIN
        DMA4(40, 40, 40, 40, " sc0 sc1") M0NEXT DMA4(41, 41, 41, 41, " sc0 sc1") M0NEXT
        DMA4(42, 42, 42, 42, " sc0 sc1") M0NEXT DMA4(43, 43, 43, 43, " sc0 sc1")
        "v_add_u32 v38, 8, v38" NL "v_add_u32 v37, 2, v37" NL
        PUBLISH
        NEXT_SLOT("0", "0xc000")
        "s_add_u32 %[step], %[step], 1" NL
        "s_cmp_lt_u32 %[step], 16" NL
        "s_cbranch_scc1 Ltop%=" NL
        "s_branch Lend%=" NL
        SPIN
        "Lend%=:" NL
        "s_waitcnt vmcnt(0)"
        : [step] "=&s"(s_step), [ld] "=&s"(s_ld), [off] "=&s"(s_off), [spin] "=&s"(s_spin), [call] "=&s"(call)
        : [g0] "s"(mt * 131072 + local_g(0, mt, sq, ntl) * 128), [g1] "s"(mt * 131072 + local_g(1, mt, sq, ntl) * 128),
          [r0] "s"(sq * 32768), [q0] "s"(sq * 4096), [aof] "s"(WS_A + mt * BLK), [cnt] "s"(WS_CNT + b * 128), [pi] "s"(pi),
          [th] "s"(tdel + PUSH_HOLDX), [tbl] "s"(IOTA), [vl] "v"(vl), [l16] "v"(lane * 16), [chka] "v"(C_DONE + (lane & 3) * 256), ARGS,
          [dl] "n"(DEADLINE), [win] "n"(XWIN), [blk4] "n"(4 * BLK), [full] "n"(C_FULL), [tmo] "n"(WS_TMO), [cbad] "n"(C_BAD)
        : "memory", "vcc", "scc", "s90", "s91", "s92", "s93", "s94", "s95", "s96", "s97", "s98", "s99",
          "v36", "v37", "v38", "v39", "v40", "v41", "v42", "v43", "v44", "v45", "v46", "v47", "v48", "v49", "v50", "v51",
          "v52", "v53", "v54", "v55");
    return;
  }
  if (wave == 2) {
    const unsigned v0 = (lane >> 4) * 8192 + (((lane & 15) ^ ((lane >> 4) << 2)) << 4);
    unsigned s_step, s_ld, s_off, s_spin, s_row;
#define W16 WIN DMA4(40, 41, 42, 43, "") M0NEXT DMA4(44, 45, 46, 47, "") M0NEXT DMA4(48, 49, 50, 51, "") M0NEXT DMA4(52, 53, 54, 55, "")
#define WBASE "s_add_u32 s98, s66, %[row]" NL "s_addc_u32 s99, s67, 0" NL "s_xor_b32 %[row], %[row], 0x40000" NL
#define WROW(dg) "s_lshl3_add_u32 %[row], s68, " dg NL "s_and_b32 %[row], %[row], 63" NL "s_lshl_b32 %[row], %[row], 19" NL \
        "s_or_b32 %[row], %[row], %[hb]" NL
#define WFIRST(h) WBASE "s_mov_b32 m0, 0xc000 + 0x4000 * " #h NL W16 "v_mov_b32 v38, 4 + 4 * " #h NL
    asm volatile(
        "v_mov_b32 v40, %[v0]" NL
        "v_xor_b32 v41, 16, v40" NL "v_add_u32 v41, 0x7c00, v41" NL
        "v_xor_b32 v42, 32, v40" NL "v_add_u32 v42, 0xf800, v42" NL
        "v_xor_b32 v43, 48, v40" NL "v_add_u32 v43, 0x17400, v43" NL
        ".irp r,44,45,46,47" NL "v_add_u32 v\\r, 0x20000, v[\\r-4]" NL ".endr" NL
        ".irp r,48,49,50,51,52,53,54,55" NL "v_add_u32 v\\r, 0x100, v[\\r-8]" NL ".endr" NL
        ARGS_IN
        WROW("%[g0]")
        "s_cmp_lt_u32 %[pi], " STR(PUSH_HOLD) NL
        "s_cbranch_scc0 Lnh%=" NL
        "s_barrier" NL
        WAIT_TIME("Lh", "%[th]")
        "Lnh%=:" NL
        WFIRST(0)
        "s_cmp_lt_u32 %[pi], " STR(PUSH_HOLD) NL
        "s_cbranch_scc1 Lnb%=" NL
        "s_barrier" NL
        "Lnb%=:" NL
        PUBLISH
        WFIRST(1) PUBLISH
        WROW("%[g1]")
        WFIRST(2) PUBLISH WFIRST(3) PUBLISH
        "v_mov_b32 v37, -1" NL
        "s_mov_b64 s[92:93], s[76:77]" NL "s_mov_b64 s[94:95], s[78:79]" NL
        "s_mov_b32 %[step], 4" NL "s_mov_b32 %[ld], 0x1c000" NL "s_mov_b32 %[spin], 0" NL
        "Ltop%=:" NL
        "s_bitcmp1_b32 %[step], 0" NL
        "s_cbranch_scc1 Lodd%=" NL
        WROW("s92")
        NEXT_DG
        "Lodd%=:" NL
        WBASE
        "s_mov_b32 m0, %[ld]" NL
        WAIT_FREE
        W16
        "v_add_u32 v38, 4, v38" NL "v_add_u32 v37, 1, v37" NL
        PUBLISH
        NEXT_SLOT("0xc000", "0x24000")
        "s_add_u32 %[step], %[step], 1" NL
        "s_cmp_lt_u32 %[step], 32" NL
        "s_cbranch_scc1 Ltop%=" NL
        "s_branch Lend%=" NL
        SPIN
        "Lend%=:" NL
        "s_waitcnt vmcnt(0)"
        : [step] "=&s"(s_step), [ld] "=&s"(s_ld), [off] "=&s"(s_off), [spin] "=&s"(s_spin), [row] "=&s"(s_row)
        : [hb] "s"(((mt >> 1) << 18) + n0 * 2), [g0] "s"(local_g(0, mt, sq, ntl)), [g1] "s"(local_g(1, mt, sq, ntl)),
          [pi] "s"(pi), [th] "s"(tdel + PUSH_HOLDX), [tbl] "s"(IOTA), [v0] "v"(v0), [chka] "v"(C_DONE + (lane & 3) * 256), ARGS,
          [win] "n"(WWIN), [full] "n"(C_FULL + 256), [cbad] "n"(C_BAD)
        : "memory", "vcc", "scc", "s92", "s93", "s94", "s95", "s96", "s97", "s98", "s99",
          "v36", "v37", "v38", "v39", "v40", "v41", "v42", "v43", "v44", "v45", "v46", "v47", "v48", "v49", "v50", "v51",
          "v52", "v53", "v54", "v55");
    return;
  }
  char* ws;
  u64 T0;
  asm volatile(ARGS_IN "s_mov_b64 %0, s[70:71]" NL "s_mov_b64 %1, s[74:75]" : "=&s"(ws), "=&s"(T0) : ARGS);
  const char* cnt = ws + WS_CNT + b * 128;
  if (wave == 3) {
    unsigned spins = 0, m = 0;
    char* dstw = nullptr;
    if (b < 8) {
      unsigned dlo = 0, dhi = 0;
#pragma unroll
      for (int i = 0; i < NR; ++i) {
        dlo = wrlane((unsigned)(size_t)p.wsrel[i], i, dlo);
        dhi = wrlane((unsigned)((size_t)p.wsrel[i] >> 32), i, dhi);
      }
      dstw = (char*)(((size_t)dhi << 32) | dlo) + WS_AM2 + (lane * 8 + b) * 128;
    }
    asm volatile("" : "+v"(dstw));
    while (uni(*(volatile LAS unsigned*)(lds + C_FULL)) < 8 && (++spins & 0x3fffff)) __builtin_amdgcn_s_sleep(8);
    if (lane == 0) st_sys((char*)cnt, call);
    while (uni(*(volatile LAS unsigned*)(lds + C_OUT)) == 0 && (++spins & 0xffffff)) __builtin_amdgcn_s_sleep(1);
    if (b < 8) {
      const char* gp = ws + WS_AM1 + (b + 8 * (lane >> 1)) * 128 + (lane & 1) * 16;
      while (true) {
        const u64 v0 = ld64_sys(gp), v1 = ld64_sys(gp + 8);
        m = max((unsigned)v0, (unsigned)v1);
        if (__all(min((unsigned)(v0 >> 32), (unsigned)(v1 >> 32)) >= call)) break;
        if ((++spins & 63) == 0 && expired(ws, call, (unsigned)T0)) { m = POISON; break; }
      }
      m = wave_umax(m);
      if (lane < NR) st64_sys(dstw, ((u64)call << 32) | m);
    }
    const char* gp = ws + WS_AM2 + lane * 128;
    unsigned tm, guess = 0;
    float inv = 0.f;
    u64 seen = 0;
    bool all;
    m = 0;
    while (true) {
      const u64 v = ld64_sys(gp);
      tm = ld_tmo(ws);
      const bool in = (unsigned)(v >> 32) >= call;
      const u64 mask = __builtin_amdgcn_ballot_w64(in);
      all = mask == ~0ull;
      if (mask != seen) m = wave_umax(in ? (unsigned)v : 0u);
      seen = mask;
      if (m != guess && __builtin_popcountll(mask) >= GUESS_MIN) {
        guess = m;
        inv = 127.0f / __uint_as_float(m);
        if (!all) *(volatile LAS u32x2*)(lds + C_GFLAG) = u32x2{__builtin_popcountll(mask) <= WT_MAX ? 4u : 3u, __float_as_uint(inv)};
      }
      if (all) break;
      if ((++spins & 63) == 0 && expired(ws, call, (unsigned)T0)) break;
    }
    *(volatile LAS u32x2*)(lds + C_GFLAG) = u32x2{(!all || m == POISON || tm == call) ? 2u : 1u, __float_as_uint(inv)};
    return;
  }

  const int cw = wave - 4, g4 = lane >> 4;
  f32x16 ac0, ac1;
  {
    const int tq = (lane & 15) >> 2, tp = lane & 3;
    const unsigned xa = (lane & 15) * 128 + ((g4 ^ ((lane & 15) >> 1) ^ ((mt & 2) << 1)) << 4);
    const unsigned wa = LDS_W + (cw >> 1) * 8192 + 256 * (8 * g4 + tq) + (((8 * (cw & 1) + (tp >> 1)) ^ ((tq << 2) | ((2 * g4) & 3))) << 4) + 8 * (tp & 1);
    unsigned s_step, s_sa, s_sw, s_da, s_dw, s_spin;
    char* xd = ws + WS_XD + (size_t)((b & ~3) * 4 + cw) * 12288;
    asm volatile(
        "s_setprio 2\n\t"
        "s_load_dword %[call], %[cnt], 0x0 glc\n\t"
        "s_barrier\n\t"
        "s_mov_b32 s93, 0x4000\n\t"
        "s_mov_b32 m0, %[m0v]\n\t"
        "s_mov_b32 %[step], 0\n\t"
        "s_mov_b32 %[sa], 0\n\t"
        "s_mov_b32 %[sw], 0\n\t"
        "s_mov_b32 %[spin], 0\n\t"
        "v_mov_b32 v224, %[xa]\n\t"
        "v_xor_b32 v225, 64, %[xa]\n\t"
        "v_mov_b32 v226, %[wa]\n\t"
        "v_xor_b32 v227, 16, %[wa]\n\t"
        "v_add_u32 v227, 0x400, v227\n\t"
        "v_xor_b32 v228, 32, v226\n\t"
        "v_xor_b32 v229, 32, v227\n\t"
        "v_xor_b32 v230, 64, v226\n\t"
        "v_xor_b32 v231, 64, v227\n\t"
        "v_xor_b32 v232, 96, v226\n\t"
        "v_xor_b32 v233, 96, v227\n\t"
        "v_mov_b32 v235, 0\n\t"
        "v_mov_b64 v[0:1], 0\n\t"
        "v_mov_b64 v[2:3], 0\n\t"
        "v_mov_b64 v[4:5], 0\n\t"
        "v_mov_b64 v[6:7], 0\n\t"
        "v_mov_b64 v[8:9], 0\n\t"
        "v_mov_b64 v[10:11], 0\n\t"
        "v_mov_b64 v[12:13], 0\n\t"
        "v_mov_b64 v[14:15], 0\n\t"
        "v_mov_b64 v[16:17], 0\n\t"
        "v_mov_b64 v[18:19], 0\n\t"
        "v_mov_b64 v[20:21], 0\n\t"
        "v_mov_b64 v[22:23], 0\n\t"
        "v_mov_b64 v[24:25], 0\n\t"
        "v_mov_b64 v[26:27], 0\n\t"
        "v_mov_b64 v[28:29], 0\n\t"
        "v_mov_b64 v[30:31], 0\n\t"
        "v_mov_b64 v[32:33], 0\n\t"
        "v_mov_b64 v[34:35], 0\n\t"
        "v_mov_b64 v[36:37], 0\n\t"
        "v_mov_b64 v[38:39], 0\n\t"
        "v_mov_b64 v[40:41], 0\n\t"
        "v_mov_b64 v[42:43], 0\n\t"
        "v_mov_b64 v[44:45], 0\n\t"
        "v_mov_b64 v[46:47], 0\n\t"
        "v_mov_b64 v[48:49], 0\n\t"
        "v_mov_b64 v[50:51], 0\n\t"
        "v_mov_b64 v[52:53], 0\n\t"
        "v_mov_b64 v[54:55], 0\n\t"
        "v_mov_b64 v[56:57], 0\n\t"
        "v_mov_b64 v[58:59], 0\n\t"
        "v_mov_b64 v[60:61], 0\n\t"
        "v_mov_b64 v[62:63], 0\n\t"
        "v_mov_b64 v[64:65], 0\n\t"
        "v_mov_b64 v[66:67], 0\n\t"
        "v_mov_b64 v[68:69], 0\n\t"
        "v_mov_b64 v[70:71], 0\n\t"
        "v_mov_b64 v[72:73], 0\n\t"
        "v_mov_b64 v[74:75], 0\n\t"
        "v_mov_b64 v[76:77], 0\n\t"
        "v_mov_b64 v[78:79], 0\n\t"
        "v_mov_b64 v[80:81], 0\n\t"
        "v_mov_b64 v[82:83], 0\n\t"
        "v_mov_b64 v[84:85], 0\n\t"
        "v_mov_b64 v[86:87], 0\n\t"
        "v_mov_b64 v[88:89], 0\n\t"
        "v_mov_b64 v[90:91], 0\n\t"
        "v_mov_b64 v[92:93], 0\n\t"
        "v_mov_b64 v[94:95], 0\n\t"
        "v_mov_b64 v[96:97], 0\n\t"
        "v_mov_b64 v[98:99], 0\n\t"
        "v_mov_b64 v[100:101], 0\n\t"
        "v_mov_b64 v[102:103], 0\n\t"
        "v_mov_b64 v[104:105], 0\n\t"
        "v_mov_b64 v[106:107], 0\n\t"
        "v_mov_b64 v[108:109], 0\n\t"
        "v_mov_b64 v[110:111], 0\n\t"
        "v_mov_b64 v[112:113], 0\n\t"
        "v_mov_b64 v[114:115], 0\n\t"
        "v_mov_b64 v[116:117], 0\n\t"
        "v_mov_b64 v[118:119], 0\n\t"
        "v_mov_b64 v[120:121], 0\n\t"
        "v_mov_b64 v[122:123], 0\n\t"
        "v_mov_b64 v[124:125], 0\n\t"
        "v_mov_b64 v[126:127], 0\n\t"
        "s_waitcnt lgkmcnt(0)\n\t"
        "s_add_u32 %[call], %[call], 1\n\t"
        "LwaitP%=:\n\t"
        "ds_read_b32 v234, %[fa]\n\t"
        "s_add_u32 %[spin], %[spin], 1\n\t"
        "s_waitcnt lgkmcnt(0)\n\t"
        "v_cmp_ge_u32 vcc, %[step], v234\n\t"
        "s_cbranch_vccz LgoP%=\n\t"
        "s_cmp_lt_u32 %[spin], 0x1000000\n\t"
        "s_cbranch_scc1 LwaitP%=\n\t"
        "LgoP%=:\n\t"
        "ds_read_b64_tr_b16 v[160:161], v226\n\t"
        "ds_read_b64_tr_b16 v[162:163], v227\n\t"
        "ds_read_b64_tr_b16 v[164:165], v228\n\t"
        "ds_read_b64_tr_b16 v[166:167], v229\n\t"
        "ds_read_b64_tr_b16 v[168:169], v230\n\t"
        "ds_read_b64_tr_b16 v[170:171], v231\n\t"
        "ds_read_b64_tr_b16 v[172:173], v232\n\t"
        "ds_read_b64_tr_b16 v[174:175], v233\n\t"
        "ds_read_b128 v[128:131], v224 offset:0\n\t"
        "ds_read_b128 v[132:135], v224 offset:2048\n\t"
        "ds_read_b128 v[136:139], v224 offset:4096\n\t"
        "ds_read_b128 v[140:143], v224 offset:6144\n\t"
        "ds_read_b128 v[144:147], v224 offset:8192\n\t"
        "ds_read_b128 v[148:151], v224 offset:10240\n\t"
        "ds_read_b128 v[152:155], v224 offset:12288\n\t"
        "ds_read_b128 v[156:159], v224 offset:14336\n\t"
        "s_add_u32 %[step], %[step], 1\n\t"
        "s_add_u32 %[sw], %[sw], 1\n\t"
        "s_cmp_eq_u32 %[sw], 6\n\t"
        "s_cselect_b32 %[sw], 0, %[sw]\n\t"
        "s_cselect_b32 %[dw], 0xfffec000, s93\n\t"
        "s_add_u32 %[sa], %[sa], 1\n\t"
        "s_cmp_eq_u32 %[sa], 3\n\t"
        "s_cselect_b32 %[sa], 0, %[sa]\n\t"
        "s_cselect_b32 %[da], 0xffff8000, s93\n\t"
        "ds_read_b32 v234, %[fa]\n\t"
        "v_add_u32 v226, %[dw], v226\n\t"
        "v_add_u32 v227, %[dw], v227\n\t"
        "v_add_u32 v228, %[dw], v228\n\t"
        "v_add_u32 v229, %[dw], v229\n\t"
        "v_add_u32 v230, %[dw], v230\n\t"
        "v_add_u32 v231, %[dw], v231\n\t"
        "v_add_u32 v232, %[dw], v232\n\t"
        "v_add_u32 v233, %[dw], v233\n\t"
        "v_add_u32 v224, %[da], v224\n\t"
        "v_add_u32 v235, 4, v235\n\t"
        "s_waitcnt lgkmcnt(0)\n\t"
        "global_load_lds_dword v235, %[tbl]\n\t"
        "v_cmp_ge_u32 vcc, %[step], v234\n\t"
        "s_cbranch_vccnz LslowP%=\n\t"
        "LfastP%=:\n\t"
        "v_mfma_f32_16x16x32_bf16 v[0:3], v[160:163], v[128:131], v[0:3]\n\t"
        "ds_read_b64_tr_b16 v[208:209], v226\n\t"
        "v_mfma_f32_16x16x32_bf16 v[4:7], v[164:167], v[128:131], v[4:7]\n\t"
        "ds_read_b64_tr_b16 v[210:211], v227\n\t"
        "v_mfma_f32_16x16x32_bf16 v[8:11], v[168:171], v[128:131], v[8:11]\n\t"
        "ds_read_b64_tr_b16 v[212:213], v228\n\t"
        "v_mfma_f32_16x16x32_bf16 v[12:15], v[172:175], v[128:131], v[12:15]\n\t"
        "ds_read_b64_tr_b16 v[214:215], v229\n\t"
        "v_mfma_f32_16x16x32_bf16 v[28:31], v[172:175], v[132:135], v[28:31]\n\t"
        "ds_read_b64_tr_b16 v[216:217], v230\n\t"
        "v_mfma_f32_16x16x32_bf16 v[24:27], v[168:171], v[132:135], v[24:27]\n\t"
        "ds_read_b64_tr_b16 v[218:219], v231\n\t"
        "v_mfma_f32_16x16x32_bf16 v[20:23], v[164:167], v[132:135], v[20:23]\n\t"
        "ds_read_b64_tr_b16 v[220:221], v232\n\t"
        "v_mfma_f32_16x16x32_bf16 v[16:19], v[160:163], v[132:135], v[16:19]\n\t"
        "ds_read_b64_tr_b16 v[222:223], v233\n\t"
        "v_mfma_f32_16x16x32_bf16 v[32:35], v[160:163], v[136:139], v[32:35]\n\t"
        "ds_read_b128 v[176:179], v225 offset:0\n\t"
        "v_mfma_f32_16x16x32_bf16 v[36:39], v[164:167], v[136:139], v[36:39]\n\t"
        "ds_read_b128 v[180:183], v225 offset:2048\n\t"
        "v_mfma_f32_16x16x32_bf16 v[40:43], v[168:171], v[136:139], v[40:43]\n\t"
        "ds_read_b128 v[184:187], v225 offset:4096\n\t"
        "v_mfma_f32_16x16x32_bf16 v[44:47], v[172:175], v[136:139], v[44:47]\n\t"
        "ds_read_b128 v[188:191], v225 offset:6144\n\t"
        "v_mfma_f32_16x16x32_bf16 v[60:63], v[172:175], v[140:143], v[60:63]\n\t"
        "ds_read_b128 v[192:195], v225 offset:8192\n\t"
        "v_mfma_f32_16x16x32_bf16 v[56:59], v[168:171], v[140:143], v[56:59]\n\t"
        "ds_read_b128 v[196:199], v225 offset:10240\n\t"
        "v_mfma_f32_16x16x32_bf16 v[52:55], v[164:167], v[140:143], v[52:55]\n\t"
        "ds_read_b128 v[200:203], v225 offset:12288\n\t"
        "v_mfma_f32_16x16x32_bf16 v[48:51], v[160:163], v[140:143], v[48:51]\n\t"
        "ds_read_b128 v[204:207], v225 offset:14336\n\t"
        "v_mfma_f32_16x16x32_bf16 v[64:67], v[160:163], v[144:147], v[64:67]\n\t"
        "s_add_u32 %[step], %[step], 1\n\t"
        "s_add_u32 %[sw], %[sw], 1\n\t"
        "s_cmp_eq_u32 %[sw], 6\n\t"
        "v_mfma_f32_16x16x32_bf16 v[68:71], v[164:167], v[144:147], v[68:71]\n\t"
        "s_cselect_b32 %[sw], 0, %[sw]\n\t"
        "s_cselect_b32 %[dw], 0xfffec000, s93\n\t"
        "v_mfma_f32_16x16x32_bf16 v[72:75], v[168:171], v[144:147], v[72:75]\n\t"
        "v_add_u32 v235, 4, v235\n\t"
        "v_mfma_f32_16x16x32_bf16 v[76:79], v[172:175], v[144:147], v[76:79]\n\t"
        "v_add_u32 v226, %[dw], v226\n\t"
        "v_mfma_f32_16x16x32_bf16 v[92:95], v[172:175], v[148:151], v[92:95]\n\t"
        "s_waitcnt lgkmcnt(0)\n\t"
        "global_load_lds_dword v235, %[tbl]\n\t"
        "v_mfma_f32_16x16x32_bf16 v[88:91], v[168:171], v[148:151], v[88:91]\n\t"
        "v_add_u32 v227, %[dw], v227\n\t"
        "v_mfma_f32_16x16x32_bf16 v[84:87], v[164:167], v[148:151], v[84:87]\n\t"
        "v_add_u32 v228, %[dw], v228\n\t"
        "v_mfma_f32_16x16x32_bf16 v[80:83], v[160:163], v[148:151], v[80:83]\n\t"
        "v_add_u32 v229, %[dw], v229\n\t"
        "v_mfma_f32_16x16x32_bf16 v[96:99], v[160:163], v[152:155], v[96:99]\n\t"
        "v_add_u32 v230, %[dw], v230\n\t"
        "v_mfma_f32_16x16x32_bf16 v[100:103], v[164:167], v[152:155], v[100:103]\n\t"
        "ds_read_b32 v234, %[fa]\n\t"
        "v_add_u32 v231, %[dw], v231\n\t"
        "v_mfma_f32_16x16x32_bf16 v[104:107], v[168:171], v[152:155], v[104:107]\n\t"
        "v_add_u32 v232, %[dw], v232\n\t"
        "v_mfma_f32_16x16x32_bf16 v[108:111], v[172:175], v[152:155], v[108:111]\n\t"
        "v_add_u32 v233, %[dw], v233\n\t"
        "v_mfma_f32_16x16x32_bf16 v[124:127], v[172:175], v[156:159], v[124:127]\n\t"
        "v_add_u32 v225, %[da], v225\n\t"
        "v_mfma_f32_16x16x32_bf16 v[120:123], v[168:171], v[156:159], v[120:123]\n\t"
        "s_cmp_eq_u32 %[step], 26\n\t"
        "v_mfma_f32_16x16x32_bf16 v[116:119], v[164:167], v[156:159], v[116:119]\n\t"
        "s_waitcnt lgkmcnt(0)\n\t"
        "v_cmp_ge_u32 vcc, %[step], v234\n\t"
        "v_mfma_f32_16x16x32_bf16 v[112:115], v[160:163], v[156:159], v[112:115]\n\t"
        "s_cbranch_scc1 Ltail%=\n\t"
        "LtopQ%=:\n\t"
        "s_cbranch_vccnz LslowQ%=\n\t"
        "v_mfma_f32_16x16x32_bf16 v[0:3], v[208:211], v[176:179], v[0:3]\n\t"
        "ds_read_b64_tr_b16 v[160:161], v226\n\t"
        "v_mfma_f32_16x16x32_bf16 v[4:7], v[212:215], v[176:179], v[4:7]\n\t"
        "ds_read_b64_tr_b16 v[162:163], v227\n\t"
        "v_mfma_f32_16x16x32_bf16 v[8:11], v[216:219], v[176:179], v[8:11]\n\t"
        "ds_read_b64_tr_b16 v[164:165], v228\n\t"
        "v_mfma_f32_16x16x32_bf16 v[12:15], v[220:223], v[176:179], v[12:15]\n\t"
        "ds_read_b64_tr_b16 v[166:167], v229\n\t"
        "v_mfma_f32_16x16x32_bf16 v[28:31], v[220:223], v[180:183], v[28:31]\n\t"
        "ds_read_b64_tr_b16 v[168:169], v230\n\t"
        "v_mfma_f32_16x16x32_bf16 v[24:27], v[216:219], v[180:183], v[24:27]\n\t"
        "ds_read_b64_tr_b16 v[170:171], v231\n\t"
        "v_mfma_f32_16x16x32_bf16 v[20:23], v[212:215], v[180:183], v[20:23]\n\t"
        "ds_read_b64_tr_b16 v[172:173], v232\n\t"
        "v_mfma_f32_16x16x32_bf16 v[16:19], v[208:211], v[180:183], v[16:19]\n\t"
        "ds_read_b64_tr_b16 v[174:175], v233\n\t"
        "v_mfma_f32_16x16x32_bf16 v[32:35], v[208:211], v[184:187], v[32:35]\n\t"
        "ds_read_b128 v[128:131], v224 offset:0\n\t"
        "v_mfma_f32_16x16x32_bf16 v[36:39], v[212:215], v[184:187], v[36:39]\n\t"
        "ds_read_b128 v[132:135], v224 offset:2048\n\t"
        "v_mfma_f32_16x16x32_bf16 v[40:43], v[216:219], v[184:187], v[40:43]\n\t"
        "ds_read_b128 v[136:139], v224 offset:4096\n\t"
        "v_mfma_f32_16x16x32_bf16 v[44:47], v[220:223], v[184:187], v[44:47]\n\t"
        "ds_read_b128 v[140:143], v224 offset:6144\n\t"
        "v_mfma_f32_16x16x32_bf16 v[60:63], v[220:223], v[188:191], v[60:63]\n\t"
        "ds_read_b128 v[144:147], v224 offset:8192\n\t"
        "v_mfma_f32_16x16x32_bf16 v[56:59], v[216:219], v[188:191], v[56:59]\n\t"
        "ds_read_b128 v[148:151], v224 offset:10240\n\t"
        "v_mfma_f32_16x16x32_bf16 v[52:55], v[212:215], v[188:191], v[52:55]\n\t"
        "ds_read_b128 v[152:155], v224 offset:12288\n\t"
        "v_mfma_f32_16x16x32_bf16 v[48:51], v[208:211], v[188:191], v[48:51]\n\t"
        "ds_read_b128 v[156:159], v224 offset:14336\n\t"
        "v_mfma_f32_16x16x32_bf16 v[64:67], v[208:211], v[192:195], v[64:67]\n\t"
        "s_add_u32 %[step], %[step], 1\n\t"
        "s_add_u32 %[sw], %[sw], 1\n\t"
        "s_cmp_eq_u32 %[sw], 6\n\t"
        "v_mfma_f32_16x16x32_bf16 v[68:71], v[212:215], v[192:195], v[68:71]\n\t"
        "s_cselect_b32 %[sw], 0, %[sw]\n\t"
        "s_cselect_b32 %[dw], 0xfffec000, s93\n\t"
        "s_add_u32 %[sa], %[sa], 1\n\t"
        "v_mfma_f32_16x16x32_bf16 v[72:75], v[216:219], v[192:195], v[72:75]\n\t"
        "s_cmp_eq_u32 %[sa], 3\n\t"
        "s_cselect_b32 %[sa], 0, %[sa]\n\t"
        "s_cselect_b32 %[da], 0xffff8000, s93\n\t"
        "v_add_u32 v235, 4, v235\n\t"
        "v_mfma_f32_16x16x32_bf16 v[76:79], v[220:223], v[192:195], v[76:79]\n\t"
        "v_add_u32 v226, %[dw], v226\n\t"
        "v_mfma_f32_16x16x32_bf16 v[92:95], v[220:223], v[196:199], v[92:95]\n\t"
        "s_waitcnt lgkmcnt(0)\n\t"
        "global_load_lds_dword v235, %[tbl]\n\t"
        "v_mfma_f32_16x16x32_bf16 v[88:91], v[216:219], v[196:199], v[88:91]\n\t"
        "v_add_u32 v227, %[dw], v227\n\t"
        "v_mfma_f32_16x16x32_bf16 v[84:87], v[212:215], v[196:199], v[84:87]\n\t"
        "v_add_u32 v228, %[dw], v228\n\t"
        "v_mfma_f32_16x16x32_bf16 v[80:83], v[208:211], v[196:199], v[80:83]\n\t"
        "v_add_u32 v229, %[dw], v229\n\t"
        "v_mfma_f32_16x16x32_bf16 v[96:99], v[208:211], v[200:203], v[96:99]\n\t"
        "v_add_u32 v230, %[dw], v230\n\t"
        "v_mfma_f32_16x16x32_bf16 v[100:103], v[212:215], v[200:203], v[100:103]\n\t"
        "ds_read_b32 v234, %[fa]\n\t"
        "v_add_u32 v231, %[dw], v231\n\t"
        "v_mfma_f32_16x16x32_bf16 v[104:107], v[216:219], v[200:203], v[104:107]\n\t"
        "v_add_u32 v232, %[dw], v232\n\t"
        "v_mfma_f32_16x16x32_bf16 v[108:111], v[220:223], v[200:203], v[108:111]\n\t"
        "v_add_u32 v233, %[dw], v233\n\t"
        "v_mfma_f32_16x16x32_bf16 v[124:127], v[220:223], v[204:207], v[124:127]\n\t"
        "v_add_u32 v224, %[da], v224\n\t"
        "v_mfma_f32_16x16x32_bf16 v[120:123], v[216:219], v[204:207], v[120:123]\n\t"
        "v_mfma_f32_16x16x32_bf16 v[116:119], v[212:215], v[204:207], v[116:119]\n\t"
        "s_waitcnt lgkmcnt(0)\n\t"
        "v_cmp_ge_u32 vcc, %[step], v234\n\t"
        "v_mfma_f32_16x16x32_bf16 v[112:115], v[208:211], v[204:207], v[112:115]\n\t"
        "s_cbranch_vccz LfastP%=\n\t"
        "LslowP%=:\n\t"
        "v_mfma_f32_16x16x32_bf16 v[0:3], v[160:163], v[128:131], v[0:3]\n\t"
        "v_mfma_f32_16x16x32_bf16 v[4:7], v[164:167], v[128:131], v[4:7]\n\t"
        "v_mfma_f32_16x16x32_bf16 v[8:11], v[168:171], v[128:131], v[8:11]\n\t"
        "v_mfma_f32_16x16x32_bf16 v[12:15], v[172:175], v[128:131], v[12:15]\n\t"
        "v_mfma_f32_16x16x32_bf16 v[28:31], v[172:175], v[132:135], v[28:31]\n\t"
        "v_mfma_f32_16x16x32_bf16 v[24:27], v[168:171], v[132:135], v[24:27]\n\t"
        "v_mfma_f32_16x16x32_bf16 v[20:23], v[164:167], v[132:135], v[20:23]\n\t"
        "v_mfma_f32_16x16x32_bf16 v[16:19], v[160:163], v[132:135], v[16:19]\n\t"
        "v_mfma_f32_16x16x32_bf16 v[32:35], v[160:163], v[136:139], v[32:35]\n\t"
        "v_mfma_f32_16x16x32_bf16 v[36:39], v[164:167], v[136:139], v[36:39]\n\t"
        "v_mfma_f32_16x16x32_bf16 v[40:43], v[168:171], v[136:139], v[40:43]\n\t"
        "v_mfma_f32_16x16x32_bf16 v[44:47], v[172:175], v[136:139], v[44:47]\n\t"
        "v_mfma_f32_16x16x32_bf16 v[60:63], v[172:175], v[140:143], v[60:63]\n\t"
        "v_mfma_f32_16x16x32_bf16 v[56:59], v[168:171], v[140:143], v[56:59]\n\t"
        "v_mfma_f32_16x16x32_bf16 v[52:55], v[164:167], v[140:143], v[52:55]\n\t"
        "v_mfma_f32_16x16x32_bf16 v[48:51], v[160:163], v[140:143], v[48:51]\n\t"
        "v_mfma_f32_16x16x32_bf16 v[64:67], v[160:163], v[144:147], v[64:67]\n\t"
        "v_mfma_f32_16x16x32_bf16 v[68:71], v[164:167], v[144:147], v[68:71]\n\t"
        "v_mfma_f32_16x16x32_bf16 v[72:75], v[168:171], v[144:147], v[72:75]\n\t"
        "v_mfma_f32_16x16x32_bf16 v[76:79], v[172:175], v[144:147], v[76:79]\n\t"
        "v_mfma_f32_16x16x32_bf16 v[92:95], v[172:175], v[148:151], v[92:95]\n\t"
        "v_mfma_f32_16x16x32_bf16 v[88:91], v[168:171], v[148:151], v[88:91]\n\t"
        "v_mfma_f32_16x16x32_bf16 v[84:87], v[164:167], v[148:151], v[84:87]\n\t"
        "v_mfma_f32_16x16x32_bf16 v[80:83], v[160:163], v[148:151], v[80:83]\n\t"
        "v_mfma_f32_16x16x32_bf16 v[96:99], v[160:163], v[152:155], v[96:99]\n\t"
        "v_mfma_f32_16x16x32_bf16 v[100:103], v[164:167], v[152:155], v[100:103]\n\t"
        "v_mfma_f32_16x16x32_bf16 v[104:107], v[168:171], v[152:155], v[104:107]\n\t"
        "v_mfma_f32_16x16x32_bf16 v[108:111], v[172:175], v[152:155], v[108:111]\n\t"
        "v_mfma_f32_16x16x32_bf16 v[124:127], v[172:175], v[156:159], v[124:127]\n\t"
        "v_mfma_f32_16x16x32_bf16 v[120:123], v[168:171], v[156:159], v[120:123]\n\t"
        "v_mfma_f32_16x16x32_bf16 v[116:119], v[164:167], v[156:159], v[116:119]\n\t"
        "v_mfma_f32_16x16x32_bf16 v[112:115], v[160:163], v[156:159], v[112:115]\n\t"
        "LwaitQ%=:\n\t"
        "ds_read_b32 v234, %[fa]\n\t"
        "s_add_u32 %[spin], %[spin], 1\n\t"
        "s_waitcnt lgkmcnt(0)\n\t"
        "v_cmp_ge_u32 vcc, %[step], v234\n\t"
        "s_cbranch_vccz LgoQ%=\n\t"
        "s_cmp_lt_u32 %[spin], 0x1000000\n\t"
        "s_cbranch_scc1 LwaitQ%=\n\t"
        "LgoQ%=:\n\t"
        "ds_read_b64_tr_b16 v[208:209], v226\n\t"
        "ds_read_b64_tr_b16 v[210:211], v227\n\t"
        "ds_read_b64_tr_b16 v[212:213], v228\n\t"
        "ds_read_b64_tr_b16 v[214:215], v229\n\t"
        "ds_read_b64_tr_b16 v[216:217], v230\n\t"
        "ds_read_b64_tr_b16 v[218:219], v231\n\t"
        "ds_read_b64_tr_b16 v[220:221], v232\n\t"
        "ds_read_b64_tr_b16 v[222:223], v233\n\t"
        "ds_read_b128 v[176:179], v225 offset:0\n\t"
        "ds_read_b128 v[180:183], v225 offset:2048\n\t"
        "ds_read_b128 v[184:187], v225 offset:4096\n\t"
        "ds_read_b128 v[188:191], v225 offset:6144\n\t"
        "ds_read_b128 v[192:195], v225 offset:8192\n\t"
        "ds_read_b128 v[196:199], v225 offset:10240\n\t"
        "ds_read_b128 v[200:203], v225 offset:12288\n\t"
        "ds_read_b128 v[204:207], v225 offset:14336\n\t"
        "s_add_u32 %[step], %[step], 1\n\t"
        "s_add_u32 %[sw], %[sw], 1\n\t"
        "s_cmp_eq_u32 %[sw], 6\n\t"
        "s_cselect_b32 %[sw], 0, %[sw]\n\t"
        "s_cselect_b32 %[dw], 0xfffec000, s93\n\t"
        "ds_read_b32 v234, %[fa]\n\t"
        "v_add_u32 v226, %[dw], v226\n\t"
        "v_add_u32 v227, %[dw], v227\n\t"
        "v_add_u32 v228, %[dw], v228\n\t"
        "v_add_u32 v229, %[dw], v229\n\t"
        "v_add_u32 v230, %[dw], v230\n\t"
        "v_add_u32 v231, %[dw], v231\n\t"
        "v_add_u32 v232, %[dw], v232\n\t"
        "v_add_u32 v233, %[dw], v233\n\t"
        "v_add_u32 v225, %[da], v225\n\t"
        "v_add_u32 v235, 4, v235\n\t"
        "s_waitcnt lgkmcnt(0)\n\t"
        "global_load_lds_dword v235, %[tbl]\n\t"
        "v_cmp_ge_u32 vcc, %[step], v234\n\t"
        "s_cmp_eq_u32 %[step], 26\n\t"
        "s_cbranch_scc1 Ltail%=\n\t"
        "s_branch LtopQ%=\n\t"
        "LslowQ%=:\n\t"
        "v_mfma_f32_16x16x32_bf16 v[0:3], v[208:211], v[176:179], v[0:3]\n\t"
        "v_mfma_f32_16x16x32_bf16 v[4:7], v[212:215], v[176:179], v[4:7]\n\t"
        "v_mfma_f32_16x16x32_bf16 v[8:11], v[216:219], v[176:179], v[8:11]\n\t"
        "v_mfma_f32_16x16x32_bf16 v[12:15], v[220:223], v[176:179], v[12:15]\n\t"
        "v_mfma_f32_16x16x32_bf16 v[28:31], v[220:223], v[180:183], v[28:31]\n\t"
        "v_mfma_f32_16x16x32_bf16 v[24:27], v[216:219], v[180:183], v[24:27]\n\t"
        "v_mfma_f32_16x16x32_bf16 v[20:23], v[212:215], v[180:183], v[20:23]\n\t"
        "v_mfma_f32_16x16x32_bf16 v[16:19], v[208:211], v[180:183], v[16:19]\n\t"
        "v_mfma_f32_16x16x32_bf16 v[32:35], v[208:211], v[184:187], v[32:35]\n\t"
        "v_mfma_f32_16x16x32_bf16 v[36:39], v[212:215], v[184:187], v[36:39]\n\t"
        "v_mfma_f32_16x16x32_bf16 v[40:43], v[216:219], v[184:187], v[40:43]\n\t"
        "v_mfma_f32_16x16x32_bf16 v[44:47], v[220:223], v[184:187], v[44:47]\n\t"
        "v_mfma_f32_16x16x32_bf16 v[60:63], v[220:223], v[188:191], v[60:63]\n\t"
        "v_mfma_f32_16x16x32_bf16 v[56:59], v[216:219], v[188:191], v[56:59]\n\t"
        "v_mfma_f32_16x16x32_bf16 v[52:55], v[212:215], v[188:191], v[52:55]\n\t"
        "v_mfma_f32_16x16x32_bf16 v[48:51], v[208:211], v[188:191], v[48:51]\n\t"
        "v_mfma_f32_16x16x32_bf16 v[64:67], v[208:211], v[192:195], v[64:67]\n\t"
        "v_mfma_f32_16x16x32_bf16 v[68:71], v[212:215], v[192:195], v[68:71]\n\t"
        "v_mfma_f32_16x16x32_bf16 v[72:75], v[216:219], v[192:195], v[72:75]\n\t"
        "v_mfma_f32_16x16x32_bf16 v[76:79], v[220:223], v[192:195], v[76:79]\n\t"
        "v_mfma_f32_16x16x32_bf16 v[92:95], v[220:223], v[196:199], v[92:95]\n\t"
        "v_mfma_f32_16x16x32_bf16 v[88:91], v[216:219], v[196:199], v[88:91]\n\t"
        "v_mfma_f32_16x16x32_bf16 v[84:87], v[212:215], v[196:199], v[84:87]\n\t"
        "v_mfma_f32_16x16x32_bf16 v[80:83], v[208:211], v[196:199], v[80:83]\n\t"
        "v_mfma_f32_16x16x32_bf16 v[96:99], v[208:211], v[200:203], v[96:99]\n\t"
        "v_mfma_f32_16x16x32_bf16 v[100:103], v[212:215], v[200:203], v[100:103]\n\t"
        "v_mfma_f32_16x16x32_bf16 v[104:107], v[216:219], v[200:203], v[104:107]\n\t"
        "v_mfma_f32_16x16x32_bf16 v[108:111], v[220:223], v[200:203], v[108:111]\n\t"
        "v_mfma_f32_16x16x32_bf16 v[124:127], v[220:223], v[204:207], v[124:127]\n\t"
        "v_mfma_f32_16x16x32_bf16 v[120:123], v[216:219], v[204:207], v[120:123]\n\t"
        "v_mfma_f32_16x16x32_bf16 v[116:119], v[212:215], v[204:207], v[116:119]\n\t"
        "v_mfma_f32_16x16x32_bf16 v[112:115], v[208:211], v[204:207], v[112:115]\n\t"
        "s_branch LwaitP%=\n\t"
        "Ltail%=:\n\t"
        "v_add_u32 v224, 0xffffc000, v224\n\t"
        "v_add_u32 v225, 0xffffc000, v225\n\t"
        "v_add_u32 v144, 0xffff8000, v226\n\t"
        "v_add_u32 v145, 0xffff8000, v227\n\t"
        "v_add_u32 v146, 0xffff8000, v228\n\t"
        "v_add_u32 v147, 0xffff8000, v229\n\t"
        "v_add_u32 v148, 0xffff8000, v230\n\t"
        "v_add_u32 v149, 0xffff8000, v231\n\t"
        "v_add_u32 v150, 0xffff8000, v232\n\t"
        "v_add_u32 v151, 0xffff8000, v233\n\t"
        "s_cbranch_vccz LgoT26%=\n\t"
        "LwaitT26%=:\n\t"
        "ds_read_b32 v234, %[fa]\n\t"
        "s_add_u32 %[spin], %[spin], 1\n\t"
        "s_waitcnt lgkmcnt(0)\n\t"
        "v_cmp_ge_u32 vcc, 26, v234\n\t"
        "s_cbranch_vccz LgoT26%=\n\t"
        "s_cmp_lt_u32 %[spin], 0x1000000\n\t"
        "s_cbranch_scc1 LwaitT26%=\n\t"
        "LgoT26%=:\n\t"
        "v_mfma_f32_16x16x32_bf16 v[0:3], v[208:211], v[176:179], v[0:3]\n\t"
        "ds_read_b64_tr_b16 v[160:161], v226 offset:0\n\t"
        "v_mfma_f32_16x16x32_bf16 v[4:7], v[212:215], v[176:179], v[4:7]\n\t"
        "ds_read_b64_tr_b16 v[162:163], v227 offset:0\n\t"
        "v_mfma_f32_16x16x32_bf16 v[8:11], v[216:219], v[176:179], v[8:11]\n\t"
        "ds_read_b128 v[128:131], v224 offset:20480\n\t"
        "v_mfma_f32_16x16x32_bf16 v[12:15], v[220:223], v[176:179], v[12:15]\n\t"
        "ds_read_b128 v[132:135], v224 offset:22528\n\t"
        "v_mfma_f32_16x16x32_bf16 v[28:31], v[220:223], v[180:183], v[28:31]\n\t"
        "ds_read_b128 v[136:139], v224 offset:24576\n\t"
        "v_mfma_f32_16x16x32_bf16 v[24:27], v[216:219], v[180:183], v[24:27]\n\t"
        "ds_read_b128 v[140:143], v224 offset:26624\n\t"
        "v_mfma_f32_16x16x32_bf16 v[20:23], v[212:215], v[180:183], v[20:23]\n\t"
        "ds_read_b32 v234, %[fa]\n\t"
        "v_mfma_f32_16x16x32_bf16 v[16:19], v[208:211], v[180:183], v[16:19]\n\t"
        "ds_read_b64_tr_b16 v[164:165], v228 offset:0\n\t"
        "v_mfma_f32_16x16x32_bf16 v[32:35], v[208:211], v[184:187], v[32:35]\n\t"
        "ds_read_b64_tr_b16 v[166:167], v229 offset:0\n\t"
        "v_mfma_f32_16x16x32_bf16 v[36:39], v[212:215], v[184:187], v[36:39]\n\t"
        "ds_read_b64_tr_b16 v[168:169], v230 offset:0\n\t"
        "v_mfma_f32_16x16x32_bf16 v[40:43], v[216:219], v[184:187], v[40:43]\n\t"
        "ds_read_b64_tr_b16 v[170:171], v231 offset:0\n\t"
        "v_mfma_f32_16x16x32_bf16 v[44:47], v[220:223], v[184:187], v[44:47]\n\t"
        "ds_read_b64_tr_b16 v[172:173], v232 offset:0\n\t"
        "v_mfma_f32_16x16x32_bf16 v[60:63], v[220:223], v[188:191], v[60:63]\n\t"
        "ds_read_b64_tr_b16 v[174:175], v233 offset:0\n\t"
        "v_mfma_f32_16x16x32_bf16 v[56:59], v[216:219], v[188:191], v[56:59]\n\t"
        "v_mfma_f32_16x16x32_bf16 v[52:55], v[212:215], v[188:191], v[52:55]\n\t"
        "v_mfma_f32_16x16x32_bf16 v[48:51], v[208:211], v[188:191], v[48:51]\n\t"
        "v_mfma_f32_16x16x32_bf16 v[64:67], v[208:211], v[192:195], v[64:67]\n\t"
        "v_mfma_f32_16x16x32_bf16 v[68:71], v[212:215], v[192:195], v[68:71]\n\t"
        "v_mfma_f32_16x16x32_bf16 v[72:75], v[216:219], v[192:195], v[72:75]\n\t"
        "v_mfma_f32_16x16x32_bf16 v[76:79], v[220:223], v[192:195], v[76:79]\n\t"
        "v_mfma_f32_16x16x32_bf16 v[92:95], v[220:223], v[196:199], v[92:95]\n\t"
        "v_mfma_f32_16x16x32_bf16 v[88:91], v[216:219], v[196:199], v[88:91]\n\t"
        "v_mfma_f32_16x16x32_bf16 v[84:87], v[212:215], v[196:199], v[84:87]\n\t"
        "v_mfma_f32_16x16x32_bf16 v[80:83], v[208:211], v[196:199], v[80:83]\n\t"
        "v_mfma_f32_16x16x32_bf16 v[96:99], v[208:211], v[200:203], v[96:99]\n\t"
        "v_mfma_f32_16x16x32_bf16 v[100:103], v[212:215], v[200:203], v[100:103]\n\t"
        "v_mfma_f32_16x16x32_bf16 v[104:107], v[216:219], v[200:203], v[104:107]\n\t"
        "v_mfma_f32_16x16x32_bf16 v[108:111], v[220:223], v[200:203], v[108:111]\n\t"
        "v_mfma_f32_16x16x32_bf16 v[124:127], v[220:223], v[204:207], v[124:127]\n\t"
        "v_mfma_f32_16x16x32_bf16 v[120:123], v[216:219], v[204:207], v[120:123]\n\t"
        "v_mfma_f32_16x16x32_bf16 v[116:119], v[212:215], v[204:207], v[116:119]\n\t"
        "s_waitcnt lgkmcnt(6)\n\t"
        "v_cmp_ge_u32 vcc, 27, v234\n\t"
        "v_mfma_f32_16x16x32_bf16 v[112:115], v[208:211], v[204:207], v[112:115]\n\t"
        "s_cbranch_vccz LgoT27%=\n\t"
        "LwaitT27%=:\n\t"
        "ds_read_b32 v234, %[fa]\n\t"
        "s_add_u32 %[spin], %[spin], 1\n\t"
        "s_waitcnt lgkmcnt(0)\n\t"
        "v_cmp_ge_u32 vcc, 27, v234\n\t"
        "s_cbranch_vccz LgoT27%=\n\t"
        "s_cmp_lt_u32 %[spin], 0x1000000\n\t"
        "s_cbranch_scc1 LwaitT27%=\n\t"
        "LgoT27%=:\n\t"
        "v_mfma_f32_16x16x32_bf16 v[32:35], v[160:163], v[128:131], v[32:35]\n\t"
        "ds_read_b64_tr_b16 v[208:209], v226 offset:16384\n\t"
        "v_mfma_f32_16x16x32_bf16 v[48:51], v[160:163], v[132:135], v[48:51]\n\t"
        "ds_read_b64_tr_b16 v[210:211], v227 offset:16384\n\t"
        "v_mfma_f32_16x16x32_bf16 v[64:67], v[160:163], v[136:139], v[64:67]\n\t"
        "ds_read_b128 v[176:179], v225 offset:20480\n\t"
        "v_mfma_f32_16x16x32_bf16 v[80:83], v[160:163], v[140:143], v[80:83]\n\t"
        "ds_read_b128 v[180:183], v225 offset:22528\n\t"
        "s_waitcnt lgkmcnt(8)\n\t"
        "v_mfma_f32_16x16x32_bf16 v[84:87], v[164:167], v[140:143], v[84:87]\n\t"
        "ds_read_b128 v[184:187], v225 offset:24576\n\t"
        "v_mfma_f32_16x16x32_bf16 v[68:71], v[164:167], v[136:139], v[68:71]\n\t"
        "ds_read_b128 v[188:191], v225 offset:26624\n\t"
        "v_mfma_f32_16x16x32_bf16 v[52:55], v[164:167], v[132:135], v[52:55]\n\t"
        "ds_read_b32 v234, %[fa]\n\t"
        "v_mfma_f32_16x16x32_bf16 v[36:39], v[164:167], v[128:131], v[36:39]\n\t"
        "ds_read_b64_tr_b16 v[212:213], v228 offset:16384\n\t"
        "s_waitcnt lgkmcnt(8)\n\t"
        "v_mfma_f32_16x16x32_bf16 v[40:43], v[168:171], v[128:131], v[40:43]\n\t"
        "ds_read_b64_tr_b16 v[214:215], v229 offset:16384\n\t"
        "v_mfma_f32_16x16x32_bf16 v[56:59], v[168:171], v[132:135], v[56:59]\n\t"
        "ds_read_b64_tr_b16 v[216:217], v230 offset:16384\n\t"
        "v_mfma_f32_16x16x32_bf16 v[72:75], v[168:171], v[136:139], v[72:75]\n\t"
        "ds_read_b64_tr_b16 v[218:219], v231 offset:16384\n\t"
        "v_mfma_f32_16x16x32_bf16 v[88:91], v[168:171], v[140:143], v[88:91]\n\t"
        "ds_read_b64_tr_b16 v[220:221], v232 offset:16384\n\t"
        "v_mfma_f32_16x16x32_bf16 v[92:95], v[172:175], v[140:143], v[92:95]\n\t"
        "ds_read_b64_tr_b16 v[222:223], v233 offset:16384\n\t"
        "v_mfma_f32_16x16x32_bf16 v[76:79], v[172:175], v[136:139], v[76:79]\n\t"
        "v_mfma_f32_16x16x32_bf16 v[60:63], v[172:175], v[132:135], v[60:63]\n\t"
        "s_waitcnt lgkmcnt(6)\n\t"
        "v_cmp_ge_u32 vcc, 28, v234\n\t"
        "v_mfma_f32_16x16x32_bf16 v[44:47], v[172:175], v[128:131], v[44:47]\n\t"
        "s_cbranch_vccz LgoT28%=\n\t"
        "LwaitT28%=:\n\t"
        "ds_read_b32 v234, %[fa]\n\t"
        "s_add_u32 %[spin], %[spin], 1\n\t"
        "s_waitcnt lgkmcnt(0)\n\t"
        "v_cmp_ge_u32 vcc, 28, v234\n\t"
        "s_cbranch_vccz LgoT28%=\n\t"
        "s_cmp_lt_u32 %[spin], 0x1000000\n\t"
        "s_cbranch_scc1 LwaitT28%=\n\t"
        "LgoT28%=:\n\t"
        "v_mfma_f32_16x16x32_bf16 v[32:35], v[208:211], v[176:179], v[32:35]\n\t"
        "ds_read_b64_tr_b16 v[160:161], v226 offset:32768\n\t"
        "v_mfma_f32_16x16x32_bf16 v[48:51], v[208:211], v[180:183], v[48:51]\n\t"
        "ds_read_b64_tr_b16 v[162:163], v227 offset:32768\n\t"
        "v_mfma_f32_16x16x32_bf16 v[64:67], v[208:211], v[184:187], v[64:67]\n\t"
        "ds_read_b128 v[128:131], v224 offset:36864\n\t"
        "v_mfma_f32_16x16x32_bf16 v[80:83], v[208:211], v[188:191], v[80:83]\n\t"
        "ds_read_b128 v[132:135], v224 offset:38912\n\t"
        "s_waitcnt lgkmcnt(8)\n\t"
        "v_mfma_f32_16x16x32_bf16 v[84:87], v[212:215], v[188:191], v[84:87]\n\t"
        "ds_read_b128 v[136:139], v224 offset:40960\n\t"
        "v_mfma_f32_16x16x32_bf16 v[68:71], v[212:215], v[184:187], v[68:71]\n\t"
        "ds_read_b128 v[140:143], v224 offset:43008\n\t"
        "v_mfma_f32_16x16x32_bf16 v[52:55], v[212:215], v[180:183], v[52:55]\n\t"
        "ds_read_b32 v234, %[fa]\n\t"
        "v_mfma_f32_16x16x32_bf16 v[36:39], v[212:215], v[176:179], v[36:39]\n\t"
        "ds_read_b64_tr_b16 v[164:165], v228 offset:32768\n\t"
        "s_waitcnt lgkmcnt(8)\n\t"
        "v_mfma_f32_16x16x32_bf16 v[40:43], v[216:219], v[176:179], v[40:43]\n\t"
        "ds_read_b64_tr_b16 v[166:167], v229 offset:32768\n\t"
        "v_mfma_f32_16x16x32_bf16 v[56:59], v[216:219], v[180:183], v[56:59]\n\t"
        "ds_read_b64_tr_b16 v[168:169], v230 offset:32768\n\t"
        "v_mfma_f32_16x16x32_bf16 v[72:75], v[216:219], v[184:187], v[72:75]\n\t"
        "ds_read_b64_tr_b16 v[170:171], v231 offset:32768\n\t"
        "v_mfma_f32_16x16x32_bf16 v[88:91], v[216:219], v[188:191], v[88:91]\n\t"
        "ds_read_b64_tr_b16 v[172:173], v232 offset:32768\n\t"
        "v_mfma_f32_16x16x32_bf16 v[92:95], v[220:223], v[188:191], v[92:95]\n\t"
        "ds_read_b64_tr_b16 v[174:175], v233 offset:32768\n\t"
        "v_mfma_f32_16x16x32_bf16 v[76:79], v[220:223], v[184:187], v[76:79]\n\t"
        "v_mfma_f32_16x16x32_bf16 v[60:63], v[220:223], v[180:183], v[60:63]\n\t"
        "s_waitcnt lgkmcnt(6)\n\t"
        "v_cmp_ge_u32 vcc, 29, v234\n\t"
        "v_mfma_f32_16x16x32_bf16 v[44:47], v[220:223], v[176:179], v[44:47]\n\t"
        "s_cbranch_vccz LgoT29%=\n\t"
        "LwaitT29%=:\n\t"
        "ds_read_b32 v234, %[fa]\n\t"
        "s_add_u32 %[spin], %[spin], 1\n\t"
        "s_waitcnt lgkmcnt(0)\n\t"
        "v_cmp_ge_u32 vcc, 29, v234\n\t"
        "s_cbranch_vccz LgoT29%=\n\t"
        "s_cmp_lt_u32 %[spin], 0x1000000\n\t"
        "s_cbranch_scc1 LwaitT29%=\n\t"
        "LgoT29%=:\n\t"
        "v_mfma_f32_16x16x32_bf16 v[32:35], v[160:163], v[128:131], v[32:35]\n\t"
        "ds_read_b64_tr_b16 v[208:209], v226 offset:49152\n\t"
        "v_mfma_f32_16x16x32_bf16 v[48:51], v[160:163], v[132:135], v[48:51]\n\t"
        "ds_read_b64_tr_b16 v[210:211], v227 offset:49152\n\t"
        "v_mfma_f32_16x16x32_bf16 v[64:67], v[160:163], v[136:139], v[64:67]\n\t"
        "ds_read_b128 v[176:179], v225 offset:36864\n\t"
        "v_mfma_f32_16x16x32_bf16 v[80:83], v[160:163], v[140:143], v[80:83]\n\t"
        "ds_read_b128 v[180:183], v225 offset:38912\n\t"
        "s_waitcnt lgkmcnt(8)\n\t"
        "v_mfma_f32_16x16x32_bf16 v[84:87], v[164:167], v[140:143], v[84:87]\n\t"
        "ds_read_b128 v[184:187], v225 offset:40960\n\t"
        "v_mfma_f32_16x16x32_bf16 v[68:71], v[164:167], v[136:139], v[68:71]\n\t"
        "ds_read_b128 v[188:191], v225 offset:43008\n\t"
        "v_mfma_f32_16x16x32_bf16 v[52:55], v[164:167], v[132:135], v[52:55]\n\t"
        "ds_read_b32 v234, %[fa]\n\t"
        "v_mfma_f32_16x16x32_bf16 v[36:39], v[164:167], v[128:131], v[36:39]\n\t"
        "ds_read_b64_tr_b16 v[212:213], v228 offset:49152\n\t"
        "s_waitcnt lgkmcnt(8)\n\t"
        "v_mfma_f32_16x16x32_bf16 v[40:43], v[168:171], v[128:131], v[40:43]\n\t"
        "ds_read_b64_tr_b16 v[214:215], v229 offset:49152\n\t"
        "v_mfma_f32_16x16x32_bf16 v[56:59], v[168:171], v[132:135], v[56:59]\n\t"
        "ds_read_b64_tr_b16 v[216:217], v230 offset:49152\n\t"
        "v_mfma_f32_16x16x32_bf16 v[72:75], v[168:171], v[136:139], v[72:75]\n\t"
        "ds_read_b64_tr_b16 v[218:219], v231 offset:49152\n\t"
        "v_mfma_f32_16x16x32_bf16 v[88:91], v[168:171], v[140:143], v[88:91]\n\t"
        "ds_read_b64_tr_b16 v[220:221], v232 offset:49152\n\t"
        "v_mfma_f32_16x16x32_bf16 v[92:95], v[172:175], v[140:143], v[92:95]\n\t"
        "ds_read_b64_tr_b16 v[222:223], v233 offset:49152\n\t"
        "v_mfma_f32_16x16x32_bf16 v[76:79], v[172:175], v[136:139], v[76:79]\n\t"
        "v_mfma_f32_16x16x32_bf16 v[60:63], v[172:175], v[132:135], v[60:63]\n\t"
        "s_waitcnt lgkmcnt(6)\n\t"
        "v_cmp_ge_u32 vcc, 30, v234\n\t"
        "v_mfma_f32_16x16x32_bf16 v[44:47], v[172:175], v[128:131], v[44:47]\n\t"
        "s_cbranch_vccz LgoTa30%=\n\t"
        "LwaitTa30%=:\n\t"
        "ds_read_b32 v234, %[fa]\n\t"
        "s_add_u32 %[spin], %[spin], 1\n\t"
        "s_waitcnt lgkmcnt(0)\n\t"
        "v_cmp_ge_u32 vcc, 30, v234\n\t"
        "s_cbranch_vccz LgoTa30%=\n\t"
        "s_cmp_lt_u32 %[spin], 0x1000000\n\t"
        "s_cbranch_scc1 LwaitTa30%=\n\t"
        "LgoTa30%=:\n\t"
        "v_mfma_f32_16x16x32_bf16 v[32:35], v[208:211], v[176:179], v[32:35]\n\t"
        "ds_read_b64_tr_b16 v[160:161], v144 offset:0\n\t"
        "v_mfma_f32_16x16x32_bf16 v[48:51], v[208:211], v[180:183], v[48:51]\n\t"
        "ds_read_b64_tr_b16 v[162:163], v145 offset:0\n\t"
        "v_mfma_f32_16x16x32_bf16 v[64:67], v[208:211], v[184:187], v[64:67]\n\t"
        "ds_read_b128 v[128:131], v224 offset:4096\n\t"
        "v_mfma_f32_16x16x32_bf16 v[80:83], v[208:211], v[188:191], v[80:83]\n\t"
        "ds_read_b128 v[132:135], v224 offset:6144\n\t"
        "s_waitcnt lgkmcnt(8)\n\t"
        "v_mfma_f32_16x16x32_bf16 v[84:87], v[212:215], v[188:191], v[84:87]\n\t"
        "ds_read_b128 v[136:139], v224 offset:8192\n\t"
        "v_mfma_f32_16x16x32_bf16 v[68:71], v[212:215], v[184:187], v[68:71]\n\t"
        "ds_read_b128 v[140:143], v224 offset:10240\n\t"
        "v_mfma_f32_16x16x32_bf16 v[52:55], v[212:215], v[180:183], v[52:55]\n\t"
        "ds_read_b32 v234, %[fa]\n\t"
        "v_mfma_f32_16x16x32_bf16 v[36:39], v[212:215], v[176:179], v[36:39]\n\t"
        "ds_read_b64_tr_b16 v[164:165], v146 offset:0\n\t"
        "s_waitcnt lgkmcnt(8)\n\t"
        "v_mfma_f32_16x16x32_bf16 v[40:43], v[216:219], v[176:179], v[40:43]\n\t"
        "ds_read_b64_tr_b16 v[166:167], v147 offset:0\n\t"
        "v_mfma_f32_16x16x32_bf16 v[56:59], v[216:219], v[180:183], v[56:59]\n\t"
        "ds_read_b64_tr_b16 v[168:169], v148 offset:0\n\t"
        "v_mfma_f32_16x16x32_bf16 v[72:75], v[216:219], v[184:187], v[72:75]\n\t"
        "ds_read_b64_tr_b16 v[170:171], v149 offset:0\n\t"
        "v_mfma_f32_16x16x32_bf16 v[88:91], v[216:219], v[188:191], v[88:91]\n\t"
        "ds_read_b64_tr_b16 v[172:173], v150 offset:0\n\t"
        "v_mfma_f32_16x16x32_bf16 v[92:95], v[220:223], v[188:191], v[92:95]\n\t"
        "ds_read_b64_tr_b16 v[174:175], v151 offset:0\n\t"
        "v_mfma_f32_16x16x32_bf16 v[76:79], v[220:223], v[184:187], v[76:79]\n\t"
        "v_mfma_f32_16x16x32_bf16 v[60:63], v[220:223], v[180:183], v[60:63]\n\t"
        "s_waitcnt lgkmcnt(6)\n\t"
        "v_cmp_ge_u32 vcc, 31, v234\n\t"
        "v_mfma_f32_16x16x32_bf16 v[44:47], v[220:223], v[176:179], v[44:47]\n\t"
        "s_cbranch_vccz LgoTa31%=\n\t"
        "LwaitTa31%=:\n\t"
        "ds_read_b32 v234, %[fa]\n\t"
        "s_add_u32 %[spin], %[spin], 1\n\t"
        "s_waitcnt lgkmcnt(0)\n\t"
        "v_cmp_ge_u32 vcc, 31, v234\n\t"
        "s_cbranch_vccz LgoTa31%=\n\t"
        "s_cmp_lt_u32 %[spin], 0x1000000\n\t"
        "s_cbranch_scc1 LwaitTa31%=\n\t"
        "LgoTa31%=:\n\t"
        "v_mfma_f32_16x16x32_bf16 v[32:35], v[160:163], v[128:131], v[32:35]\n\t"
        "ds_read_b64_tr_b16 v[208:209], v144 offset:16384\n\t"
        "v_mfma_f32_16x16x32_bf16 v[48:51], v[160:163], v[132:135], v[48:51]\n\t"
        "ds_read_b64_tr_b16 v[210:211], v145 offset:16384\n\t"
        "v_mfma_f32_16x16x32_bf16 v[64:67], v[160:163], v[136:139], v[64:67]\n\t"
        "ds_read_b128 v[176:179], v225 offset:4096\n\t"
        "v_mfma_f32_16x16x32_bf16 v[80:83], v[160:163], v[140:143], v[80:83]\n\t"
        "ds_read_b128 v[180:183], v225 offset:6144\n\t"
        "s_waitcnt lgkmcnt(8)\n\t"
        "v_mfma_f32_16x16x32_bf16 v[84:87], v[164:167], v[140:143], v[84:87]\n\t"
        "ds_read_b128 v[184:187], v225 offset:8192\n\t"
        "v_mfma_f32_16x16x32_bf16 v[68:71], v[164:167], v[136:139], v[68:71]\n\t"
        "ds_read_b128 v[188:191], v225 offset:10240\n\t"
        "v_mfma_f32_16x16x32_bf16 v[52:55], v[164:167], v[132:135], v[52:55]\n\t"
        "ds_read_b64_tr_b16 v[212:213], v146 offset:16384\n\t"
        "v_mfma_f32_16x16x32_bf16 v[36:39], v[164:167], v[128:131], v[36:39]\n\t"
        "ds_read_b64_tr_b16 v[214:215], v147 offset:16384\n\t"
        "s_waitcnt lgkmcnt(8)\n\t"
        "v_mfma_f32_16x16x32_bf16 v[40:43], v[168:171], v[128:131], v[40:43]\n\t"
        "ds_read_b64_tr_b16 v[216:217], v148 offset:16384\n\t"
        "v_mfma_f32_16x16x32_bf16 v[56:59], v[168:171], v[132:135], v[56:59]\n\t"
        "ds_read_b64_tr_b16 v[218:219], v149 offset:16384\n\t"
        "v_mfma_f32_16x16x32_bf16 v[72:75], v[168:171], v[136:139], v[72:75]\n\t"
        "ds_read_b64_tr_b16 v[220:221], v150 offset:16384\n\t"
        "v_mfma_f32_16x16x32_bf16 v[88:91], v[168:171], v[140:143], v[88:91]\n\t"
        "ds_read_b64_tr_b16 v[222:223], v151 offset:16384\n\t"
        "v_mfma_f32_16x16x32_bf16 v[92:95], v[172:175], v[140:143], v[92:95]\n\t"
        "v_mfma_f32_16x16x32_bf16 v[76:79], v[172:175], v[136:139], v[76:79]\n\t"
        "v_mfma_f32_16x16x32_bf16 v[60:63], v[172:175], v[132:135], v[60:63]\n\t"
        "s_waitcnt lgkmcnt(6)\n\t"
        "v_mfma_f32_16x16x32_bf16 v[44:47], v[172:175], v[128:131], v[44:47]\n\t"
        "v_mfma_f32_16x16x32_bf16 v[32:35], v[208:211], v[176:179], v[32:35]\n\t"
        "ds_read_b64_tr_b16 v[160:161], v226 offset:0\n\t"
        "v_mfma_f32_16x16x32_bf16 v[48:51], v[208:211], v[180:183], v[48:51]\n\t"
        "ds_read_b64_tr_b16 v[162:163], v227 offset:0\n\t"
        "v_mfma_f32_16x16x32_bf16 v[64:67], v[208:211], v[184:187], v[64:67]\n\t"
        "ds_read_b128 v[128:131], v224 offset:28672\n\t"
        "v_mfma_f32_16x16x32_bf16 v[80:83], v[208:211], v[188:191], v[80:83]\n\t"
        "ds_read_b128 v[132:135], v224 offset:30720\n\t"
        "s_waitcnt lgkmcnt(8)\n\t"
        "v_mfma_f32_16x16x32_bf16 v[84:87], v[212:215], v[188:191], v[84:87]\n\t"
        "ds_read_b128 v[136:139], v224 offset:16384\n\t"
        "v_mfma_f32_16x16x32_bf16 v[68:71], v[212:215], v[184:187], v[68:71]\n\t"
        "ds_read_b128 v[140:143], v224 offset:18432\n\t"
        "v_mfma_f32_16x16x32_bf16 v[52:55], v[212:215], v[180:183], v[52:55]\n\t"
        "ds_read_b64_tr_b16 v[164:165], v228 offset:0\n\t"
        "v_mfma_f32_16x16x32_bf16 v[36:39], v[212:215], v[176:179], v[36:39]\n\t"
        "ds_read_b64_tr_b16 v[166:167], v229 offset:0\n\t"
        "s_waitcnt lgkmcnt(8)\n\t"
        "v_mfma_f32_16x16x32_bf16 v[40:43], v[216:219], v[176:179], v[40:43]\n\t"
        "ds_read_b64_tr_b16 v[168:169], v230 offset:0\n\t"
        "v_mfma_f32_16x16x32_bf16 v[56:59], v[216:219], v[180:183], v[56:59]\n\t"
        "ds_read_b64_tr_b16 v[170:171], v231 offset:0\n\t"
        "v_mfma_f32_16x16x32_bf16 v[72:75], v[216:219], v[184:187], v[72:75]\n\t"
        "ds_read_b64_tr_b16 v[172:173], v232 offset:0\n\t"
        "v_mfma_f32_16x16x32_bf16 v[88:91], v[216:219], v[188:191], v[88:91]\n\t"
        "ds_read_b64_tr_b16 v[174:175], v233 offset:0\n\t"
        "v_mfma_f32_16x16x32_bf16 v[92:95], v[220:223], v[188:191], v[92:95]\n\t"
        "v_mfma_f32_16x16x32_bf16 v[76:79], v[220:223], v[184:187], v[76:79]\n\t"
        "v_mfma_f32_16x16x32_bf16 v[60:63], v[220:223], v[180:183], v[60:63]\n\t"
        "s_waitcnt lgkmcnt(6)\n\t"
        "v_mfma_f32_16x16x32_bf16 v[44:47], v[220:223], v[176:179], v[44:47]\n\t"
        "v_mfma_f32_16x16x32_bf16 v[96:99], v[160:163], v[128:131], v[96:99]\n\t"
        "ds_read_b64_tr_b16 v[208:209], v226 offset:16384\n\t"
        "v_mfma_f32_16x16x32_bf16 v[112:115], v[160:163], v[132:135], v[112:115]\n\t"
        "ds_read_b64_tr_b16 v[210:211], v227 offset:16384\n\t"
        "v_mfma_f32_16x16x32_bf16 v[0:3], v[160:163], v[136:139], v[0:3]\n\t"
        "ds_read_b128 v[176:179], v225 offset:28672\n\t"
        "v_mfma_f32_16x16x32_bf16 v[16:19], v[160:163], v[140:143], v[16:19]\n\t"
        "ds_read_b128 v[180:183], v225 offset:30720\n\t"
        "v_cvt_pk_f16_f32 v192, v32, v33\n\t"
        "s_waitcnt lgkmcnt(8)\n\t"
        "v_mfma_f32_16x16x32_bf16 v[20:23], v[164:167], v[140:143], v[20:23]\n\t"
        "ds_read_b128 v[184:187], v225 offset:16384\n\t"
        "v_cvt_pk_f16_f32 v193, v34, v35\n\t"
        "v_mfma_f32_16x16x32_bf16 v[4:7], v[164:167], v[136:139], v[4:7]\n\t"
        "ds_read_b128 v[188:191], v225 offset:18432\n\t"
        "v_cvt_pk_f16_f32 v194, v36, v37\n\t"
        "v_mfma_f32_16x16x32_bf16 v[116:119], v[164:167], v[132:135], v[116:119]\n\t"
        "ds_read_b64_tr_b16 v[212:213], v228 offset:16384\n\t"
        "v_cvt_pk_f16_f32 v195, v38, v39\n\t"
        "v_mfma_f32_16x16x32_bf16 v[100:103], v[164:167], v[128:131], v[100:103]\n\t"
        "ds_read_b64_tr_b16 v[214:215], v229 offset:16384\n\t"
        "global_store_dwordx4 %[vo0], v[192:195], %[xo1] offset:0 sc0 sc1\n\t"
        "s_waitcnt lgkmcnt(8)\n\t"
        "v_mfma_f32_16x16x32_bf16 v[104:107], v[168:171], v[128:131], v[104:107]\n\t"
        "ds_read_b64_tr_b16 v[216:217], v230 offset:16384\n\t"
        "v_cvt_pk_f16_f32 v196, v40, v41\n\t"
        "v_mfma_f32_16x16x32_bf16 v[120:123], v[168:171], v[132:135], v[120:123]\n\t"
        "ds_read_b64_tr_b16 v[218:219], v231 offset:16384\n\t"
        "v_cvt_pk_f16_f32 v197, v42, v43\n\t"
        "v_mfma_f32_16x16x32_bf16 v[8:11], v[168:171], v[136:139], v[8:11]\n\t"
        "ds_read_b64_tr_b16 v[220:221], v232 offset:16384\n\t"
        "v_cvt_pk_f16_f32 v198, v44, v45\n\t"
        "v_mfma_f32_16x16x32_bf16 v[24:27], v[168:171], v[140:143], v[24:27]\n\t"
        "ds_read_b64_tr_b16 v[222:223], v233 offset:16384\n\t"
        "v_cvt_pk_f16_f32 v199, v46, v47\n\t"
        "v_mfma_f32_16x16x32_bf16 v[28:31], v[172:175], v[140:143], v[28:31]\n\t"
        "global_store_dwordx4 %[vo0], v[196:199], %[xo1] offset:1024 sc0 sc1\n\t"
        "v_cvt_pk_f16_f32 v200, v48, v49\n\t"
        "v_mfma_f32_16x16x32_bf16 v[12:15], v[172:175], v[136:139], v[12:15]\n\t"
        "v_cvt_pk_f16_f32 v201, v50, v51\n\t"
        "v_cvt_pk_f16_f32 v202, v52, v53\n\t"
        "v_mfma_f32_16x16x32_bf16 v[124:127], v[172:175], v[132:135], v[124:127]\n\t"
        "v_cvt_pk_f16_f32 v203, v54, v55\n\t"
        "global_store_dwordx4 %[vo0], v[200:203], %[xo1] offset:2048 sc0 sc1\n\t"
        "s_waitcnt lgkmcnt(6)\n\t"
        "v_mfma_f32_16x16x32_bf16 v[108:111], v[172:175], v[128:131], v[108:111]\n\t"
        "v_cvt_pk_f16_f32 v204, v56, v57\n\t"
        "v_cvt_pk_f16_f32 v205, v58, v59\n\t"
        "v_mfma_f32_16x16x32_bf16 v[96:99], v[208:211], v[176:179], v[96:99]\n\t"
        "ds_read_b64_tr_b16 v[160:161], v226 offset:32768\n\t"
        "v_cvt_pk_f16_f32 v206, v60, v61\n\t"
        "v_mfma_f32_16x16x32_bf16 v[112:115], v[208:211], v[180:183], v[112:115]\n\t"
        "ds_read_b64_tr_b16 v[162:163], v227 offset:32768\n\t"
        "v_cvt_pk_f16_f32 v207, v62, v63\n\t"
        "v_mfma_f32_16x16x32_bf16 v[0:3], v[208:211], v[184:187], v[0:3]\n\t"
        "ds_read_b128 v[128:131], v224 offset:45056\n\t"
        "global_store_dwordx4 %[vo0], v[204:207], %[xo1] offset:3072 sc0 sc1\n\t"
        "v_mfma_f32_16x16x32_bf16 v[16:19], v[208:211], v[188:191], v[16:19]\n\t"
        "ds_read_b128 v[132:135], v224 offset:47104\n\t"
        "v_cvt_pk_f16_f32 v192, v64, v65\n\t"
        "s_waitcnt lgkmcnt(8)\n\t"
        "v_mfma_f32_16x16x32_bf16 v[20:23], v[212:215], v[188:191], v[20:23]\n\t"
        "ds_read_b128 v[136:139], v224 offset:32768\n\t"
        "v_cvt_pk_f16_f32 v193, v66, v67\n\t"
        "v_mfma_f32_16x16x32_bf16 v[4:7], v[212:215], v[184:187], v[4:7]\n\t"
        "ds_read_b128 v[140:143], v224 offset:34816\n\t"
        "v_cvt_pk_f16_f32 v194, v68, v69\n\t"
        "v_mfma_f32_16x16x32_bf16 v[116:119], v[212:215], v[180:183], v[116:119]\n\t"
        "ds_read_b64_tr_b16 v[164:165], v228 offset:32768\n\t"
        "v_cvt_pk_f16_f32 v195, v70, v71\n\t"
        "v_mfma_f32_16x16x32_bf16 v[100:103], v[212:215], v[176:179], v[100:103]\n\t"
        "ds_read_b64_tr_b16 v[166:167], v229 offset:32768\n\t"
        "global_store_dwordx4 %[vo0], v[192:195], %[xo2] offset:0 sc0 sc1\n\t"
        "s_waitcnt lgkmcnt(8)\n\t"
        "v_mfma_f32_16x16x32_bf16 v[104:107], v[216:219], v[176:179], v[104:107]\n\t"
        "ds_read_b64_tr_b16 v[168:169], v230 offset:32768\n\t"
        "v_cvt_pk_f16_f32 v196, v72, v73\n\t"
        "v_mfma_f32_16x16x32_bf16 v[120:123], v[216:219], v[180:183], v[120:123]\n\t"
        "ds_read_b64_tr_b16 v[170:171], v231 offset:32768\n\t"
        "v_cvt_pk_f16_f32 v197, v74, v75\n\t"
        "v_mfma_f32_16x16x32_bf16 v[8:11], v[216:219], v[184:187], v[8:11]\n\t"
        "ds_read_b64_tr_b16 v[172:173], v232 offset:32768\n\t"
        "v_cvt_pk_f16_f32 v198, v76, v77\n\t"
        "v_mfma_f32_16x16x32_bf16 v[24:27], v[216:219], v[188:191], v[24:27]\n\t"
        "ds_read_b64_tr_b16 v[174:175], v233 offset:32768\n\t"
        "v_cvt_pk_f16_f32 v199, v78, v79\n\t"
        "v_mfma_f32_16x16x32_bf16 v[28:31], v[220:223], v[188:191], v[28:31]\n\t"
        "global_store_dwordx4 %[vo0], v[196:199], %[xo2] offset:1024 sc0 sc1\n\t"
        "v_cvt_pk_f16_f32 v200, v80, v81\n\t"
        "v_mfma_f32_16x16x32_bf16 v[12:15], v[220:223], v[184:187], v[12:15]\n\t"
        "v_cvt_pk_f16_f32 v201, v82, v83\n\t"
        "v_cvt_pk_f16_f32 v202, v84, v85\n\t"
        "v_mfma_f32_16x16x32_bf16 v[124:127], v[220:223], v[180:183], v[124:127]\n\t"
        "v_cvt_pk_f16_f32 v203, v86, v87\n\t"
        "global_store_dwordx4 %[vo0], v[200:203], %[xo2] offset:2048 sc0 sc1\n\t"
        "s_waitcnt lgkmcnt(6)\n\t"
        "v_mfma_f32_16x16x32_bf16 v[108:111], v[220:223], v[176:179], v[108:111]\n\t"
        "v_cvt_pk_f16_f32 v204, v88, v89\n\t"
        "v_cvt_pk_f16_f32 v205, v90, v91\n\t"
        "v_mfma_f32_16x16x32_bf16 v[96:99], v[160:163], v[128:131], v[96:99]\n\t"
        "ds_read_b64_tr_b16 v[208:209], v226 offset:49152\n\t"
        "v_cvt_pk_f16_f32 v206, v92, v93\n\t"
        "v_mfma_f32_16x16x32_bf16 v[112:115], v[160:163], v[132:135], v[112:115]\n\t"
        "ds_read_b64_tr_b16 v[210:211], v227 offset:49152\n\t"
        "v_cvt_pk_f16_f32 v207, v94, v95\n\t"
        "v_mfma_f32_16x16x32_bf16 v[0:3], v[160:163], v[136:139], v[0:3]\n\t"
        "ds_read_b128 v[176:179], v225 offset:45056\n\t"
        "global_store_dwordx4 %[vo0], v[204:207], %[xo2] offset:3072 sc0 sc1\n\t"
        "v_mfma_f32_16x16x32_bf16 v[16:19], v[160:163], v[140:143], v[16:19]\n\t"
        "ds_read_b128 v[180:183], v225 offset:47104\n\t"
        "v_max_f32 v152, |v32|, |v33|\n\t"
        "s_waitcnt lgkmcnt(8)\n\t"
        "v_mfma_f32_16x16x32_bf16 v[20:23], v[164:167], v[140:143], v[20:23]\n\t"
        "ds_read_b128 v[184:187], v225 offset:32768\n\t"
        "v_max3_f32 v152, v152, |v34|, |v35|\n\t"
        "v_mfma_f32_16x16x32_bf16 v[4:7], v[164:167], v[136:139], v[4:7]\n\t"
        "ds_read_b128 v[188:191], v225 offset:34816\n\t"
        "v_max3_f32 v152, v152, |v36|, |v37|\n\t"
        "v_mfma_f32_16x16x32_bf16 v[116:119], v[164:167], v[132:135], v[116:119]\n\t"
        "ds_read_b64_tr_b16 v[212:213], v228 offset:49152\n\t"
        "v_max3_f32 v152, v152, |v38|, |v39|\n\t"
        "v_mfma_f32_16x16x32_bf16 v[100:103], v[164:167], v[128:131], v[100:103]\n\t"
        "ds_read_b64_tr_b16 v[214:215], v229 offset:49152\n\t"
        "v_max3_f32 v152, v152, |v40|, |v41|\n\t"
        "s_waitcnt lgkmcnt(8)\n\t"
        "v_mfma_f32_16x16x32_bf16 v[104:107], v[168:171], v[128:131], v[104:107]\n\t"
        "ds_read_b64_tr_b16 v[216:217], v230 offset:49152\n\t"
        "v_max3_f32 v152, v152, |v42|, |v43|\n\t"
        "v_mfma_f32_16x16x32_bf16 v[120:123], v[168:171], v[132:135], v[120:123]\n\t"
        "ds_read_b64_tr_b16 v[218:219], v231 offset:49152\n\t"
        "v_max3_f32 v152, v152, |v44|, |v45|\n\t"
        "v_mfma_f32_16x16x32_bf16 v[8:11], v[168:171], v[136:139], v[8:11]\n\t"
        "ds_read_b64_tr_b16 v[220:221], v232 offset:49152\n\t"
        "v_max3_f32 v152, v152, |v46|, |v47|\n\t"
        "v_mfma_f32_16x16x32_bf16 v[24:27], v[168:171], v[140:143], v[24:27]\n\t"
        "ds_read_b64_tr_b16 v[222:223], v233 offset:49152\n\t"
        "v_max3_f32 v152, v152, |v48|, |v49|\n\t"
        "v_mfma_f32_16x16x32_bf16 v[28:31], v[172:175], v[140:143], v[28:31]\n\t"
        "v_max3_f32 v152, v152, |v50|, |v51|\n\t"
        "v_max3_f32 v152, v152, |v52|, |v53|\n\t"
        "v_mfma_f32_16x16x32_bf16 v[12:15], v[172:175], v[136:139], v[12:15]\n\t"
        "v_max3_f32 v152, v152, |v54|, |v55|\n\t"
        "v_max3_f32 v152, v152, |v56|, |v57|\n\t"
        "v_mfma_f32_16x16x32_bf16 v[124:127], v[172:175], v[132:135], v[124:127]\n\t"
        "v_max3_f32 v152, v152, |v58|, |v59|\n\t"
        "v_max3_f32 v152, v152, |v60|, |v61|\n\t"
        "s_waitcnt lgkmcnt(6)\n\t"
        "v_mfma_f32_16x16x32_bf16 v[108:111], v[172:175], v[128:131], v[108:111]\n\t"
        "v_max3_f32 v152, v152, |v62|, |v63|\n\t"
        "v_max3_f32 v152, v152, |v64|, |v65|\n\t"
        "v_mfma_f32_16x16x32_bf16 v[96:99], v[208:211], v[176:179], v[96:99]\n\t"
        "ds_read_b64_tr_b16 v[160:161], v144 offset:0\n\t"
        "v_max3_f32 v152, v152, |v66|, |v67|\n\t"
        "v_mfma_f32_16x16x32_bf16 v[112:115], v[208:211], v[180:183], v[112:115]\n\t"
        "ds_read_b64_tr_b16 v[162:163], v145 offset:0\n\t"
        "v_max3_f32 v152, v152, |v68|, |v69|\n\t"
        "v_mfma_f32_16x16x32_bf16 v[0:3], v[208:211], v[184:187], v[0:3]\n\t"
        "ds_read_b128 v[128:131], v224 offset:12288\n\t"
        "v_max3_f32 v152, v152, |v70|, |v71|\n\t"
        "v_mfma_f32_16x16x32_bf16 v[16:19], v[208:211], v[188:191], v[16:19]\n\t"
        "ds_read_b128 v[132:135], v224 offset:14336\n\t"
        "v_max3_f32 v152, v152, |v72|, |v73|\n\t"
        "s_waitcnt lgkmcnt(8)\n\t"
        "v_mfma_f32_16x16x32_bf16 v[20:23], v[212:215], v[188:191], v[20:23]\n\t"
        "ds_read_b128 v[136:139], v224 offset:0\n\t"
        "v_max3_f32 v152, v152, |v74|, |v75|\n\t"
        "v_mfma_f32_16x16x32_bf16 v[4:7], v[212:215], v[184:187], v[4:7]\n\t"
        "ds_read_b128 v[140:143], v224 offset:2048\n\t"
        "v_max3_f32 v152, v152, |v76|, |v77|\n\t"
        "v_mfma_f32_16x16x32_bf16 v[116:119], v[212:215], v[180:183], v[116:119]\n\t"
        "ds_read_b64_tr_b16 v[164:165], v146 offset:0\n\t"
        "v_max3_f32 v152, v152, |v78|, |v79|\n\t"
        "v_mfma_f32_16x16x32_bf16 v[100:103], v[212:215], v[176:179], v[100:103]\n\t"
        "ds_read_b64_tr_b16 v[166:167], v147 offset:0\n\t"
        "v_max3_f32 v152, v152, |v80|, |v81|\n\t"
        "s_waitcnt lgkmcnt(8)\n\t"
        "v_mfma_f32_16x16x32_bf16 v[104:107], v[216:219], v[176:179], v[104:107]\n\t"
        "ds_read_b64_tr_b16 v[168:169], v148 offset:0\n\t"
        "v_max3_f32 v152, v152, |v82|, |v83|\n\t"
        "v_mfma_f32_16x16x32_bf16 v[120:123], v[216:219], v[180:183], v[120:123]\n\t"
        "ds_read_b64_tr_b16 v[170:171], v149 offset:0\n\t"
        "v_max3_f32 v152, v152, |v84|, |v85|\n\t"
        "v_mfma_f32_16x16x32_bf16 v[8:11], v[216:219], v[184:187], v[8:11]\n\t"
        "ds_read_b64_tr_b16 v[172:173], v150 offset:0\n\t"
        "v_max3_f32 v152, v152, |v86|, |v87|\n\t"
        "v_mfma_f32_16x16x32_bf16 v[24:27], v[216:219], v[188:191], v[24:27]\n\t"
        "ds_read_b64_tr_b16 v[174:175], v151 offset:0\n\t"
        "v_max3_f32 v152, v152, |v88|, |v89|\n\t"
        "v_mfma_f32_16x16x32_bf16 v[28:31], v[220:223], v[188:191], v[28:31]\n\t"
        "v_max3_f32 v152, v152, |v90|, |v91|\n\t"
        "v_max3_f32 v152, v152, |v92|, |v93|\n\t"
        "v_mfma_f32_16x16x32_bf16 v[12:15], v[220:223], v[184:187], v[12:15]\n\t"
        "v_max3_f32 v152, v152, |v94|, |v95|\n\t"
        "v_mfma_f32_16x16x32_bf16 v[124:127], v[220:223], v[180:183], v[124:127]\n\t"
        "s_waitcnt lgkmcnt(6)\n\t"
        "v_mfma_f32_16x16x32_bf16 v[108:111], v[220:223], v[176:179], v[108:111]\n\t"
        "v_mfma_f32_16x16x32_bf16 v[96:99], v[160:163], v[128:131], v[96:99]\n\t"
        "ds_read_b64_tr_b16 v[208:209], v144 offset:16384\n\t"
        "v_mfma_f32_16x16x32_bf16 v[112:115], v[160:163], v[132:135], v[112:115]\n\t"
        "ds_read_b64_tr_b16 v[210:211], v145 offset:16384\n\t"
        "v_mfma_f32_16x16x32_bf16 v[0:3], v[160:163], v[136:139], v[0:3]\n\t"
        "ds_read_b128 v[176:179], v225 offset:12288\n\t"
        "v_mfma_f32_16x16x32_bf16 v[16:19], v[160:163], v[140:143], v[16:19]\n\t"
        "ds_read_b128 v[180:183], v225 offset:14336\n\t"
        "s_waitcnt lgkmcnt(8)\n\t"
        "v_mfma_f32_16x16x32_bf16 v[20:23], v[164:167], v[140:143], v[20:23]\n\t"
        "ds_read_b128 v[184:187], v225 offset:0\n\t"
        "v_mfma_f32_16x16x32_bf16 v[4:7], v[164:167], v[136:139], v[4:7]\n\t"
        "ds_read_b128 v[188:191], v225 offset:2048\n\t"
        "v_mfma_f32_16x16x32_bf16 v[116:119], v[164:167], v[132:135], v[116:119]\n\t"
        "ds_read_b64_tr_b16 v[212:213], v146 offset:16384\n\t"
        "v_mfma_f32_16x16x32_bf16 v[100:103], v[164:167], v[128:131], v[100:103]\n\t"
        "ds_read_b64_tr_b16 v[214:215], v147 offset:16384\n\t"
        "s_waitcnt lgkmcnt(8)\n\t"
        "v_mfma_f32_16x16x32_bf16 v[104:107], v[168:171], v[128:131], v[104:107]\n\t"
        "ds_read_b64_tr_b16 v[216:217], v148 offset:16384\n\t"
        "v_mfma_f32_16x16x32_bf16 v[120:123], v[168:171], v[132:135], v[120:123]\n\t"
        "ds_read_b64_tr_b16 v[218:219], v149 offset:16384\n\t"
        "v_mfma_f32_16x16x32_bf16 v[8:11], v[168:171], v[136:139], v[8:11]\n\t"
        "ds_read_b64_tr_b16 v[220:221], v150 offset:16384\n\t"
        "v_mfma_f32_16x16x32_bf16 v[24:27], v[168:171], v[140:143], v[24:27]\n\t"
        "ds_read_b64_tr_b16 v[222:223], v151 offset:16384\n\t"
        "v_mfma_f32_16x16x32_bf16 v[28:31], v[172:175], v[140:143], v[28:31]\n\t"
        "v_mfma_f32_16x16x32_bf16 v[12:15], v[172:175], v[136:139], v[12:15]\n\t"
        "v_mfma_f32_16x16x32_bf16 v[124:127], v[172:175], v[132:135], v[124:127]\n\t"
        "s_waitcnt lgkmcnt(6)\n\t"
        "v_mfma_f32_16x16x32_bf16 v[108:111], v[172:175], v[128:131], v[108:111]\n\t"
        "s_waitcnt lgkmcnt(0)\n\t"
        "v_mfma_f32_16x16x32_bf16 v[96:99], v[208:211], v[176:179], v[96:99]\n\t"
        "v_mfma_f32_16x16x32_bf16 v[100:103], v[212:215], v[176:179], v[100:103]\n\t"
        "v_mfma_f32_16x16x32_bf16 v[104:107], v[216:219], v[176:179], v[104:107]\n\t"
        "v_mfma_f32_16x16x32_bf16 v[108:111], v[220:223], v[176:179], v[108:111]\n\t"
        "v_mfma_f32_16x16x32_bf16 v[112:115], v[208:211], v[180:183], v[112:115]\n\t"
        "v_mfma_f32_16x16x32_bf16 v[116:119], v[212:215], v[180:183], v[116:119]\n\t"
        "v_mfma_f32_16x16x32_bf16 v[120:123], v[216:219], v[180:183], v[120:123]\n\t"
        "v_mfma_f32_16x16x32_bf16 v[124:127], v[220:223], v[180:183], v[124:127]\n\t"
        "v_mfma_f32_16x16x32_bf16 v[0:3], v[208:211], v[184:187], v[0:3]\n\t"
        "v_cvt_pk_f16_f32 v192, v96, v97\n\t"
        "v_cvt_pk_f16_f32 v193, v98, v99\n\t"
        "v_mfma_f32_16x16x32_bf16 v[4:7], v[212:215], v[184:187], v[4:7]\n\t"
        "v_cvt_pk_f16_f32 v194, v100, v101\n\t"
        "v_cvt_pk_f16_f32 v195, v102, v103\n\t"
        "global_store_dwordx4 %[vo0], v[192:195], %[xo3] offset:0 sc0 sc1\n\t"
        "v_mfma_f32_16x16x32_bf16 v[8:11], v[216:219], v[184:187], v[8:11]\n\t"
        "v_cvt_pk_f16_f32 v196, v104, v105\n\t"
        "v_cvt_pk_f16_f32 v197, v106, v107\n\t"
        "v_mfma_f32_16x16x32_bf16 v[12:15], v[220:223], v[184:187], v[12:15]\n\t"
        "v_cvt_pk_f16_f32 v198, v108, v109\n\t"
        "v_cvt_pk_f16_f32 v199, v110, v111\n\t"
        "global_store_dwordx4 %[vo0], v[196:199], %[xo3] offset:1024 sc0 sc1\n\t"
        "v_mfma_f32_16x16x32_bf16 v[16:19], v[208:211], v[188:191], v[16:19]\n\t"
        "v_cvt_pk_f16_f32 v200, v112, v113\n\t"
        "v_cvt_pk_f16_f32 v201, v114, v115\n\t"
        "v_mfma_f32_16x16x32_bf16 v[20:23], v[212:215], v[188:191], v[20:23]\n\t"
        "v_cvt_pk_f16_f32 v202, v116, v117\n\t"
        "v_cvt_pk_f16_f32 v203, v118, v119\n\t"
        "global_store_dwordx4 %[vo0], v[200:203], %[xo3] offset:2048 sc0 sc1\n\t"
        "v_mfma_f32_16x16x32_bf16 v[24:27], v[216:219], v[188:191], v[24:27]\n\t"
        "v_cvt_pk_f16_f32 v204, v120, v121\n\t"
        "v_cvt_pk_f16_f32 v205, v122, v123\n\t"
        "v_mfma_f32_16x16x32_bf16 v[28:31], v[220:223], v[188:191], v[28:31]\n\t"
        "v_cvt_pk_f16_f32 v206, v124, v125\n\t"
        "v_cvt_pk_f16_f32 v207, v126, v127\n\t"
        "global_store_dwordx4 %[vo0], v[204:207], %[xo3] offset:3072 sc0 sc1\n\t"
        "s_mov_b32 %[step], 32\n\t"
        "v_max3_f32 v152, v152, |v96|, |v97|\n\t"
        "v_max3_f32 v152, v152, |v98|, |v99|\n\t"
        "v_max3_f32 v152, v152, |v100|, |v101|\n\t"
        "v_max3_f32 v152, v152, |v102|, |v103|\n\t"
        "v_max3_f32 v152, v152, |v104|, |v105|\n\t"
        "v_max3_f32 v152, v152, |v106|, |v107|\n\t"
        "v_max3_f32 v152, v152, |v108|, |v109|\n\t"
        "v_max3_f32 v152, v152, |v110|, |v111|\n\t"
        "v_max3_f32 v152, v152, |v112|, |v113|\n\t"
        "v_max3_f32 v152, v152, |v114|, |v115|\n\t"
        "v_max3_f32 v152, v152, |v116|, |v117|\n\t"
        "v_max3_f32 v152, v152, |v118|, |v119|\n\t"
        "v_max3_f32 v152, v152, |v120|, |v121|\n\t"
        "v_max3_f32 v152, v152, |v122|, |v123|\n\t"
        "v_max3_f32 v152, v152, |v124|, |v125|\n\t"
        "v_max3_f32 v152, v152, |v126|, |v127|\n\t"
        "s_nop 1\n\t"
        "v_max_u32_dpp v152, v152, v152 row_shr:1 row_mask:0xf bank_mask:0xf bound_ctrl:1\n\t"
        "s_nop 1\n\t"
        "v_max_u32_dpp v152, v152, v152 row_shr:2 row_mask:0xf bank_mask:0xf bound_ctrl:1\n\t"
        "s_nop 1\n\t"
        "v_max_u32_dpp v152, v152, v152 row_shr:4 row_mask:0xf bank_mask:0xf bound_ctrl:1\n\t"
        "s_nop 1\n\t"
        "v_max_u32_dpp v152, v152, v152 row_shr:8 row_mask:0xf bank_mask:0xf bound_ctrl:1\n\t"
        "s_nop 1\n\t"
        "v_max_u32_dpp v152, v152, v152 row_bcast:15 row_mask:0xa bank_mask:0xf\n\t"
        "s_nop 1\n\t"
        "v_max_u32_dpp v152, v152, v152 row_bcast:31 row_mask:0xc bank_mask:0xf\n\t"
        "s_nop 0\n\t"
        "v_readlane_b32 %[sa], v152, 63\n\t"
        "v_mov_b32 v154, %[call]\n\t"
        "v_mov_b32 v155, 1.0\n\t"
        "s_lshr_b32 %[sa], %[sa], 23\n\t"
        "s_sub_u32 %[sw], %[sa], 123\n\t"
        "s_cmp_lt_u32 %[sw], 19\n\t"
        "s_cbranch_scc1 Lokb%=\n\t"
        "s_max_u32 %[sa], %[sa], 15\n\t"
        "s_min_u32 %[sa], %[sa], 254\n\t"
        "s_sub_u32 s94, 268, %[sa]\n\t"
        "s_lshl_b32 s94, s94, 23\n\t"
        "s_sub_u32 %[sa], %[sa], 14\n\t"
        "s_lshl_b32 %[sa], %[sa], 23\n\t"
        "v_mov_b32 v155, %[sa]\n\t"
        "s_waitcnt vmcnt(0)\n\t"
        "v_pk_mul_f32 v[32:33], v[32:33], s[94:95] op_sel_hi:[1,0]\n\t"
        "v_pk_mul_f32 v[34:35], v[34:35], s[94:95] op_sel_hi:[1,0]\n\t"
        "v_pk_mul_f32 v[36:37], v[36:37], s[94:95] op_sel_hi:[1,0]\n\t"
        "v_pk_mul_f32 v[38:39], v[38:39], s[94:95] op_sel_hi:[1,0]\n\t"
        "v_pk_mul_f32 v[40:41], v[40:41], s[94:95] op_sel_hi:[1,0]\n\t"
        "v_pk_mul_f32 v[42:43], v[42:43], s[94:95] op_sel_hi:[1,0]\n\t"
        "v_pk_mul_f32 v[44:45], v[44:45], s[94:95] op_sel_hi:[1,0]\n\t"
        "v_pk_mul_f32 v[46:47], v[46:47], s[94:95] op_sel_hi:[1,0]\n\t"
        "v_pk_mul_f32 v[48:49], v[48:49], s[94:95] op_sel_hi:[1,0]\n\t"
        "v_pk_mul_f32 v[50:51], v[50:51], s[94:95] op_sel_hi:[1,0]\n\t"
        "v_pk_mul_f32 v[52:53], v[52:53], s[94:95] op_sel_hi:[1,0]\n\t"
        "v_pk_mul_f32 v[54:55], v[54:55], s[94:95] op_sel_hi:[1,0]\n\t"
        "v_pk_mul_f32 v[56:57], v[56:57], s[94:95] op_sel_hi:[1,0]\n\t"
        "v_pk_mul_f32 v[58:59], v[58:59], s[94:95] op_sel_hi:[1,0]\n\t"
        "v_pk_mul_f32 v[60:61], v[60:61], s[94:95] op_sel_hi:[1,0]\n\t"
        "v_pk_mul_f32 v[62:63], v[62:63], s[94:95] op_sel_hi:[1,0]\n\t"
        "v_cvt_pk_f16_f32 v192, v32, v33\n\t"
        "v_cvt_pk_f16_f32 v193, v34, v35\n\t"
        "v_cvt_pk_f16_f32 v194, v36, v37\n\t"
        "v_cvt_pk_f16_f32 v195, v38, v39\n\t"
        "global_store_dwordx4 %[vo0], v[192:195], %[xo1] offset:0 sc0 sc1\n\t"
        "v_cvt_pk_f16_f32 v196, v40, v41\n\t"
        "v_cvt_pk_f16_f32 v197, v42, v43\n\t"
        "v_cvt_pk_f16_f32 v198, v44, v45\n\t"
        "v_cvt_pk_f16_f32 v199, v46, v47\n\t"
        "global_store_dwordx4 %[vo0], v[196:199], %[xo1] offset:1024 sc0 sc1\n\t"
        "v_cvt_pk_f16_f32 v200, v48, v49\n\t"
        "v_cvt_pk_f16_f32 v201, v50, v51\n\t"
        "v_cvt_pk_f16_f32 v202, v52, v53\n\t"
        "v_cvt_pk_f16_f32 v203, v54, v55\n\t"
        "global_store_dwordx4 %[vo0], v[200:203], %[xo1] offset:2048 sc0 sc1\n\t"
        "v_cvt_pk_f16_f32 v204, v56, v57\n\t"
        "v_cvt_pk_f16_f32 v205, v58, v59\n\t"
        "v_cvt_pk_f16_f32 v206, v60, v61\n\t"
        "v_cvt_pk_f16_f32 v207, v62, v63\n\t"
        "global_store_dwordx4 %[vo0], v[204:207], %[xo1] offset:3072 sc0 sc1\n\t"
        "v_pk_mul_f32 v[64:65], v[64:65], s[94:95] op_sel_hi:[1,0]\n\t"
        "v_pk_mul_f32 v[66:67], v[66:67], s[94:95] op_sel_hi:[1,0]\n\t"
        "v_pk_mul_f32 v[68:69], v[68:69], s[94:95] op_sel_hi:[1,0]\n\t"
        "v_pk_mul_f32 v[70:71], v[70:71], s[94:95] op_sel_hi:[1,0]\n\t"
        "v_pk_mul_f32 v[72:73], v[72:73], s[94:95] op_sel_hi:[1,0]\n\t"
        "v_pk_mul_f32 v[74:75], v[74:75], s[94:95] op_sel_hi:[1,0]\n\t"
        "v_pk_mul_f32 v[76:77], v[76:77], s[94:95] op_sel_hi:[1,0]\n\t"
        "v_pk_mul_f32 v[78:79], v[78:79], s[94:95] op_sel_hi:[1,0]\n\t"
        "v_pk_mul_f32 v[80:81], v[80:81], s[94:95] op_sel_hi:[1,0]\n\t"
        "v_pk_mul_f32 v[82:83], v[82:83], s[94:95] op_sel_hi:[1,0]\n\t"
        "v_pk_mul_f32 v[84:85], v[84:85], s[94:95] op_sel_hi:[1,0]\n\t"
        "v_pk_mul_f32 v[86:87], v[86:87], s[94:95] op_sel_hi:[1,0]\n\t"
        "v_pk_mul_f32 v[88:89], v[88:89], s[94:95] op_sel_hi:[1,0]\n\t"
        "v_pk_mul_f32 v[90:91], v[90:91], s[94:95] op_sel_hi:[1,0]\n\t"
        "v_pk_mul_f32 v[92:93], v[92:93], s[94:95] op_sel_hi:[1,0]\n\t"
        "v_pk_mul_f32 v[94:95], v[94:95], s[94:95] op_sel_hi:[1,0]\n\t"
        "v_cvt_pk_f16_f32 v192, v64, v65\n\t"
        "v_cvt_pk_f16_f32 v193, v66, v67\n\t"
        "v_cvt_pk_f16_f32 v194, v68, v69\n\t"
        "v_cvt_pk_f16_f32 v195, v70, v71\n\t"
        "global_store_dwordx4 %[vo0], v[192:195], %[xo2] offset:0 sc0 sc1\n\t"
        "v_cvt_pk_f16_f32 v196, v72, v73\n\t"
        "v_cvt_pk_f16_f32 v197, v74, v75\n\t"
        "v_cvt_pk_f16_f32 v198, v76, v77\n\t"
        "v_cvt_pk_f16_f32 v199, v78, v79\n\t"
        "global_store_dwordx4 %[vo0], v[196:199], %[xo2] offset:1024 sc0 sc1\n\t"
        "v_cvt_pk_f16_f32 v200, v80, v81\n\t"
        "v_cvt_pk_f16_f32 v201, v82, v83\n\t"
        "v_cvt_pk_f16_f32 v202, v84, v85\n\t"
        "v_cvt_pk_f16_f32 v203, v86, v87\n\t"
        "global_store_dwordx4 %[vo0], v[200:203], %[xo2] offset:2048 sc0 sc1\n\t"
        "v_cvt_pk_f16_f32 v204, v88, v89\n\t"
        "v_cvt_pk_f16_f32 v205, v90, v91\n\t"
        "v_cvt_pk_f16_f32 v206, v92, v93\n\t"
        "v_cvt_pk_f16_f32 v207, v94, v95\n\t"
        "global_store_dwordx4 %[vo0], v[204:207], %[xo2] offset:3072 sc0 sc1\n\t"
        "v_pk_mul_f32 v[96:97], v[96:97], s[94:95] op_sel_hi:[1,0]\n\t"
        "v_pk_mul_f32 v[98:99], v[98:99], s[94:95] op_sel_hi:[1,0]\n\t"
        "v_pk_mul_f32 v[100:101], v[100:101], s[94:95] op_sel_hi:[1,0]\n\t"
        "v_pk_mul_f32 v[102:103], v[102:103], s[94:95] op_sel_hi:[1,0]\n\t"
        "v_pk_mul_f32 v[104:105], v[104:105], s[94:95] op_sel_hi:[1,0]\n\t"
        "v_pk_mul_f32 v[106:107], v[106:107], s[94:95] op_sel_hi:[1,0]\n\t"
        "v_pk_mul_f32 v[108:109], v[108:109], s[94:95] op_sel_hi:[1,0]\n\t"
        "v_pk_mul_f32 v[110:111], v[110:111], s[94:95] op_sel_hi:[1,0]\n\t"
        "v_pk_mul_f32 v[112:113], v[112:113], s[94:95] op_sel_hi:[1,0]\n\t"
        "v_pk_mul_f32 v[114:115], v[114:115], s[94:95] op_sel_hi:[1,0]\n\t"
        "v_pk_mul_f32 v[116:117], v[116:117], s[94:95] op_sel_hi:[1,0]\n\t"
        "v_pk_mul_f32 v[118:119], v[118:119], s[94:95] op_sel_hi:[1,0]\n\t"
        "v_pk_mul_f32 v[120:121], v[120:121], s[94:95] op_sel_hi:[1,0]\n\t"
        "v_pk_mul_f32 v[122:123], v[122:123], s[94:95] op_sel_hi:[1,0]\n\t"
        "v_pk_mul_f32 v[124:125], v[124:125], s[94:95] op_sel_hi:[1,0]\n\t"
        "v_pk_mul_f32 v[126:127], v[126:127], s[94:95] op_sel_hi:[1,0]\n\t"
        "v_cvt_pk_f16_f32 v192, v96, v97\n\t"
        "v_cvt_pk_f16_f32 v193, v98, v99\n\t"
        "v_cvt_pk_f16_f32 v194, v100, v101\n\t"
        "v_cvt_pk_f16_f32 v195, v102, v103\n\t"
        "global_store_dwordx4 %[vo0], v[192:195], %[xo3] offset:0 sc0 sc1\n\t"
        "v_cvt_pk_f16_f32 v196, v104, v105\n\t"
        "v_cvt_pk_f16_f32 v197, v106, v107\n\t"
        "v_cvt_pk_f16_f32 v198, v108, v109\n\t"
        "v_cvt_pk_f16_f32 v199, v110, v111\n\t"
        "global_store_dwordx4 %[vo0], v[196:199], %[xo3] offset:1024 sc0 sc1\n\t"
        "v_cvt_pk_f16_f32 v200, v112, v113\n\t"
        "v_cvt_pk_f16_f32 v201, v114, v115\n\t"
        "v_cvt_pk_f16_f32 v202, v116, v117\n\t"
        "v_cvt_pk_f16_f32 v203, v118, v119\n\t"
        "global_store_dwordx4 %[vo0], v[200:203], %[xo3] offset:2048 sc0 sc1\n\t"
        "v_cvt_pk_f16_f32 v204, v120, v121\n\t"
        "v_cvt_pk_f16_f32 v205, v122, v123\n\t"
        "v_cvt_pk_f16_f32 v206, v124, v125\n\t"
        "v_cvt_pk_f16_f32 v207, v126, v127\n\t"
        "global_store_dwordx4 %[vo0], v[204:207], %[xo3] offset:3072 sc0 sc1\n\t"
        "Lokb%=:\n\t"
        "v_add_u32 v182, 0x1000, %[vo0]\n\t"
        "v_add_u32 v183, 0x2000, %[vo0]\n\t"
        "v_mov_b32 v179, %[call]\n\t"
        "v_mov_b32 v185, 0\n\t"
        "v_mov_b32 v186, %[cbad]\n\t"
        "s_mov_b32 %[sa], 0\n\t"
        "s_waitcnt vmcnt(4)\n\t"
        "s_mov_b64 exec, 3\n\t"
        "global_store_dwordx2 %[fd], v[154:155], %[xfo] sc0 sc1\n\t"
        "s_mov_b64 exec, -1\n\t"
        "s_waitcnt vmcnt(1)\n\t"
        "s_mov_b64 exec, 4\n\t"
        "global_store_dwordx2 %[fd], v[154:155], %[xfo] sc0 sc1\n\t"
        "s_mov_b64 exec, -1\n\t"
        "Lx%=:\n\t"
        "s_load_dwordx2 s[80:81], %[xfi], 0x0 glc\n\t"
        "s_load_dwordx2 s[82:83], %[xfi], 0x80 glc\n\t"
        "s_load_dwordx2 s[84:85], %[xfi], 0x100 glc\n\t"
        "s_memrealtime s[86:87]\n\t"
        "s_waitcnt lgkmcnt(0)\n\t"
        "s_mov_b32 %[da], 0\n\t"
        "s_bitcmp1_b32 %[sa], 0\n\t"
        "s_cbranch_scc1 Lxla0%=\n\t"
        "s_cmp_ge_u32 s80, %[call]\n\t"
        "s_cbranch_scc0 Lxla0%=\n\t"
        "global_load_dwordx4 v[128:131], %[vo0], %[xi] offset:0 sc0 sc1\n\t"
        "global_load_dwordx4 v[132:135], %[vo0], %[xi] offset:1024 sc0 sc1\n\t"
        "global_load_dwordx4 v[136:139], %[vo0], %[xi] offset:2048 sc0 sc1\n\t"
        "global_load_dwordx4 v[140:143], %[vo0], %[xi] offset:3072 sc0 sc1\n\t"
        "s_bitset1_b32 %[da], 0\n\t"
        "Lxla0%=:\n\t"
        "s_bitcmp1_b32 %[sa], 1\n\t"
        "s_cbranch_scc1 Lxla1%=\n\t"
        "s_cmp_ge_u32 s82, %[call]\n\t"
        "s_cbranch_scc0 Lxla1%=\n\t"
        "global_load_dwordx4 v[144:147], v182, %[xi] offset:0 sc0 sc1\n\t"
        "global_load_dwordx4 v[148:151], v182, %[xi] offset:1024 sc0 sc1\n\t"
        "global_load_dwordx4 v[152:155], v182, %[xi] offset:2048 sc0 sc1\n\t"
        "global_load_dwordx4 v[156:159], v182, %[xi] offset:3072 sc0 sc1\n\t"
        "s_bitset1_b32 %[da], 1\n\t"
        "Lxla1%=:\n\t"
        "s_bitcmp1_b32 %[sa], 2\n\t"
        "s_cbranch_scc1 Lxla2%=\n\t"
        "s_cmp_ge_u32 s84, %[call]\n\t"
        "s_cbranch_scc0 Lxla2%=\n\t"
        "global_load_dwordx4 v[160:163], v183, %[xi] offset:0 sc0 sc1\n\t"
        "global_load_dwordx4 v[164:167], v183, %[xi] offset:1024 sc0 sc1\n\t"
        "global_load_dwordx4 v[168:171], v183, %[xi] offset:2048 sc0 sc1\n\t"
        "global_load_dwordx4 v[172:175], v183, %[xi] offset:3072 sc0 sc1\n\t"
        "s_bitset1_b32 %[da], 2\n\t"
        "Lxla2%=:\n\t"
        "s_or_b32 %[sa], %[sa], %[da]\n\t"
        "s_cmp_eq_u32 %[sa], 7\n\t"
        "s_cbranch_scc1 Lxa%=\n\t"
        "s_sub_u32 s86, s86, %[t0]\n\t"
        "s_cmp_lt_u32 s86, 60000000\n\t"
        "s_cbranch_scc1 Lx%=\n\t"
        "s_mov_b32 %[spin], 0x1000000\n\t"
        "s_waitcnt vmcnt(0)\n\t"
        "s_branch Lxe%=\n\t"
        "Lxa%=:\n\t"
        "ds_read_b32 v184, v186\n\t"
        "s_bcnt1_i32_b32 %[sw], %[da]\n\t"
        "s_waitcnt vmcnt(8)\n\t"
        "s_cmp_eq_u32 %[sw], 1\n\t"
        "s_cbranch_scc0 Lxw%=\n\t"
        "s_waitcnt vmcnt(4)\n\t"
        "Lxw%=:\n\t"
        "s_bitcmp1_b32 %[da], 0\n\t"
        "s_cbranch_scc1 Lxf0%=\n\t"
        "v_fma_mix_f32 v0, v128, s81, v0 op_sel:[0,0,0] op_sel_hi:[1,0,0]\n\t"
        "v_fma_mix_f32 v1, v128, s81, v1 op_sel:[1,0,0] op_sel_hi:[1,0,0]\n\t"
        "v_fma_mix_f32 v2, v129, s81, v2 op_sel:[0,0,0] op_sel_hi:[1,0,0]\n\t"
        "v_fma_mix_f32 v3, v129, s81, v3 op_sel:[1,0,0] op_sel_hi:[1,0,0]\n\t"
        "v_fma_mix_f32 v4, v130, s81, v4 op_sel:[0,0,0] op_sel_hi:[1,0,0]\n\t"
        "v_fma_mix_f32 v5, v130, s81, v5 op_sel:[1,0,0] op_sel_hi:[1,0,0]\n\t"
        "v_fma_mix_f32 v6, v131, s81, v6 op_sel:[0,0,0] op_sel_hi:[1,0,0]\n\t"
        "v_fma_mix_f32 v7, v131, s81, v7 op_sel:[1,0,0] op_sel_hi:[1,0,0]\n\t"
        "v_fma_mix_f32 v8, v132, s81, v8 op_sel:[0,0,0] op_sel_hi:[1,0,0]\n\t"
        "v_fma_mix_f32 v9, v132, s81, v9 op_sel:[1,0,0] op_sel_hi:[1,0,0]\n\t"
        "v_fma_mix_f32 v10, v133, s81, v10 op_sel:[0,0,0] op_sel_hi:[1,0,0]\n\t"
        "v_fma_mix_f32 v11, v133, s81, v11 op_sel:[1,0,0] op_sel_hi:[1,0,0]\n\t"
        "v_fma_mix_f32 v12, v134, s81, v12 op_sel:[0,0,0] op_sel_hi:[1,0,0]\n\t"
        "v_fma_mix_f32 v13, v134, s81, v13 op_sel:[1,0,0] op_sel_hi:[1,0,0]\n\t"
        "v_fma_mix_f32 v14, v135, s81, v14 op_sel:[0,0,0] op_sel_hi:[1,0,0]\n\t"
        "v_fma_mix_f32 v15, v135, s81, v15 op_sel:[1,0,0] op_sel_hi:[1,0,0]\n\t"
        "v_fma_mix_f32 v16, v136, s81, v16 op_sel:[0,0,0] op_sel_hi:[1,0,0]\n\t"
        "v_fma_mix_f32 v17, v136, s81, v17 op_sel:[1,0,0] op_sel_hi:[1,0,0]\n\t"
        "v_fma_mix_f32 v18, v137, s81, v18 op_sel:[0,0,0] op_sel_hi:[1,0,0]\n\t"
        "v_fma_mix_f32 v19, v137, s81, v19 op_sel:[1,0,0] op_sel_hi:[1,0,0]\n\t"
        "v_fma_mix_f32 v20, v138, s81, v20 op_sel:[0,0,0] op_sel_hi:[1,0,0]\n\t"
        "v_fma_mix_f32 v21, v138, s81, v21 op_sel:[1,0,0] op_sel_hi:[1,0,0]\n\t"
        "v_fma_mix_f32 v22, v139, s81, v22 op_sel:[0,0,0] op_sel_hi:[1,0,0]\n\t"
        "v_fma_mix_f32 v23, v139, s81, v23 op_sel:[1,0,0] op_sel_hi:[1,0,0]\n\t"
        "v_fma_mix_f32 v24, v140, s81, v24 op_sel:[0,0,0] op_sel_hi:[1,0,0]\n\t"
        "v_fma_mix_f32 v25, v140, s81, v25 op_sel:[1,0,0] op_sel_hi:[1,0,0]\n\t"
        "v_fma_mix_f32 v26, v141, s81, v26 op_sel:[0,0,0] op_sel_hi:[1,0,0]\n\t"
        "v_fma_mix_f32 v27, v141, s81, v27 op_sel:[1,0,0] op_sel_hi:[1,0,0]\n\t"
        "v_fma_mix_f32 v28, v142, s81, v28 op_sel:[0,0,0] op_sel_hi:[1,0,0]\n\t"
        "v_fma_mix_f32 v29, v142, s81, v29 op_sel:[1,0,0] op_sel_hi:[1,0,0]\n\t"
        "v_fma_mix_f32 v30, v143, s81, v30 op_sel:[0,0,0] op_sel_hi:[1,0,0]\n\t"
        "v_fma_mix_f32 v31, v143, s81, v31 op_sel:[1,0,0] op_sel_hi:[1,0,0]\n\t"
        "Lxf0%=:\n\t"
        "s_bitcmp1_b32 %[da], 1\n\t"
        "s_cbranch_scc1 Lxf1%=\n\t"
        "v_fma_mix_f32 v0, v144, s83, v0 op_sel:[0,0,0] op_sel_hi:[1,0,0]\n\t"
        "v_fma_mix_f32 v1, v144, s83, v1 op_sel:[1,0,0] op_sel_hi:[1,0,0]\n\t"
        "v_fma_mix_f32 v2, v145, s83, v2 op_sel:[0,0,0] op_sel_hi:[1,0,0]\n\t"
        "v_fma_mix_f32 v3, v145, s83, v3 op_sel:[1,0,0] op_sel_hi:[1,0,0]\n\t"
        "v_fma_mix_f32 v4, v146, s83, v4 op_sel:[0,0,0] op_sel_hi:[1,0,0]\n\t"
        "v_fma_mix_f32 v5, v146, s83, v5 op_sel:[1,0,0] op_sel_hi:[1,0,0]\n\t"
        "v_fma_mix_f32 v6, v147, s83, v6 op_sel:[0,0,0] op_sel_hi:[1,0,0]\n\t"
        "v_fma_mix_f32 v7, v147, s83, v7 op_sel:[1,0,0] op_sel_hi:[1,0,0]\n\t"
        "v_fma_mix_f32 v8, v148, s83, v8 op_sel:[0,0,0] op_sel_hi:[1,0,0]\n\t"
        "v_fma_mix_f32 v9, v148, s83, v9 op_sel:[1,0,0] op_sel_hi:[1,0,0]\n\t"
        "v_fma_mix_f32 v10, v149, s83, v10 op_sel:[0,0,0] op_sel_hi:[1,0,0]\n\t"
        "v_fma_mix_f32 v11, v149, s83, v11 op_sel:[1,0,0] op_sel_hi:[1,0,0]\n\t"
        "v_fma_mix_f32 v12, v150, s83, v12 op_sel:[0,0,0] op_sel_hi:[1,0,0]\n\t"
        "v_fma_mix_f32 v13, v150, s83, v13 op_sel:[1,0,0] op_sel_hi:[1,0,0]\n\t"
        "v_fma_mix_f32 v14, v151, s83, v14 op_sel:[0,0,0] op_sel_hi:[1,0,0]\n\t"
        "v_fma_mix_f32 v15, v151, s83, v15 op_sel:[1,0,0] op_sel_hi:[1,0,0]\n\t"
        "v_fma_mix_f32 v16, v152, s83, v16 op_sel:[0,0,0] op_sel_hi:[1,0,0]\n\t"
        "v_fma_mix_f32 v17, v152, s83, v17 op_sel:[1,0,0] op_sel_hi:[1,0,0]\n\t"
        "v_fma_mix_f32 v18, v153, s83, v18 op_sel:[0,0,0] op_sel_hi:[1,0,0]\n\t"
        "v_fma_mix_f32 v19, v153, s83, v19 op_sel:[1,0,0] op_sel_hi:[1,0,0]\n\t"
        "v_fma_mix_f32 v20, v154, s83, v20 op_sel:[0,0,0] op_sel_hi:[1,0,0]\n\t"
        "v_fma_mix_f32 v21, v154, s83, v21 op_sel:[1,0,0] op_sel_hi:[1,0,0]\n\t"
        "v_fma_mix_f32 v22, v155, s83, v22 op_sel:[0,0,0] op_sel_hi:[1,0,0]\n\t"
        "v_fma_mix_f32 v23, v155, s83, v23 op_sel:[1,0,0] op_sel_hi:[1,0,0]\n\t"
        "v_fma_mix_f32 v24, v156, s83, v24 op_sel:[0,0,0] op_sel_hi:[1,0,0]\n\t"
        "v_fma_mix_f32 v25, v156, s83, v25 op_sel:[1,0,0] op_sel_hi:[1,0,0]\n\t"
        "v_fma_mix_f32 v26, v157, s83, v26 op_sel:[0,0,0] op_sel_hi:[1,0,0]\n\t"
        "v_fma_mix_f32 v27, v157, s83, v27 op_sel:[1,0,0] op_sel_hi:[1,0,0]\n\t"
        "v_fma_mix_f32 v28, v158, s83, v28 op_sel:[0,0,0] op_sel_hi:[1,0,0]\n\t"
        "v_fma_mix_f32 v29, v158, s83, v29 op_sel:[1,0,0] op_sel_hi:[1,0,0]\n\t"
        "v_fma_mix_f32 v30, v159, s83, v30 op_sel:[0,0,0] op_sel_hi:[1,0,0]\n\t"
        "v_fma_mix_f32 v31, v159, s83, v31 op_sel:[1,0,0] op_sel_hi:[1,0,0]\n\t"
        "Lxf1%=:\n\t"
        "s_bitcmp1_b32 %[da], 2\n\t"
        "s_cbranch_scc1 Lxf2%=\n\t"
        "v_fma_mix_f32 v0, v160, s85, v0 op_sel:[0,0,0] op_sel_hi:[1,0,0]\n\t"
        "v_fma_mix_f32 v1, v160, s85, v1 op_sel:[1,0,0] op_sel_hi:[1,0,0]\n\t"
        "v_fma_mix_f32 v2, v161, s85, v2 op_sel:[0,0,0] op_sel_hi:[1,0,0]\n\t"
        "v_fma_mix_f32 v3, v161, s85, v3 op_sel:[1,0,0] op_sel_hi:[1,0,0]\n\t"
        "v_fma_mix_f32 v4, v162, s85, v4 op_sel:[0,0,0] op_sel_hi:[1,0,0]\n\t"
        "v_fma_mix_f32 v5, v162, s85, v5 op_sel:[1,0,0] op_sel_hi:[1,0,0]\n\t"
        "v_fma_mix_f32 v6, v163, s85, v6 op_sel:[0,0,0] op_sel_hi:[1,0,0]\n\t"
        "v_fma_mix_f32 v7, v163, s85, v7 op_sel:[1,0,0] op_sel_hi:[1,0,0]\n\t"
        "v_fma_mix_f32 v8, v164, s85, v8 op_sel:[0,0,0] op_sel_hi:[1,0,0]\n\t"
        "v_fma_mix_f32 v9, v164, s85, v9 op_sel:[1,0,0] op_sel_hi:[1,0,0]\n\t"
        "v_fma_mix_f32 v10, v165, s85, v10 op_sel:[0,0,0] op_sel_hi:[1,0,0]\n\t"
        "v_fma_mix_f32 v11, v165, s85, v11 op_sel:[1,0,0] op_sel_hi:[1,0,0]\n\t"
        "v_fma_mix_f32 v12, v166, s85, v12 op_sel:[0,0,0] op_sel_hi:[1,0,0]\n\t"
        "v_fma_mix_f32 v13, v166, s85, v13 op_sel:[1,0,0] op_sel_hi:[1,0,0]\n\t"
        "v_fma_mix_f32 v14, v167, s85, v14 op_sel:[0,0,0] op_sel_hi:[1,0,0]\n\t"
        "v_fma_mix_f32 v15, v167, s85, v15 op_sel:[1,0,0] op_sel_hi:[1,0,0]\n\t"
        "v_fma_mix_f32 v16, v168, s85, v16 op_sel:[0,0,0] op_sel_hi:[1,0,0]\n\t"
        "v_fma_mix_f32 v17, v168, s85, v17 op_sel:[1,0,0] op_sel_hi:[1,0,0]\n\t"
        "v_fma_mix_f32 v18, v169, s85, v18 op_sel:[0,0,0] op_sel_hi:[1,0,0]\n\t"
        "v_fma_mix_f32 v19, v169, s85, v19 op_sel:[1,0,0] op_sel_hi:[1,0,0]\n\t"
        "v_fma_mix_f32 v20, v170, s85, v20 op_sel:[0,0,0] op_sel_hi:[1,0,0]\n\t"
        "v_fma_mix_f32 v21, v170, s85, v21 op_sel:[1,0,0] op_sel_hi:[1,0,0]\n\t"
        "v_fma_mix_f32 v22, v171, s85, v22 op_sel:[0,0,0] op_sel_hi:[1,0,0]\n\t"
        "v_fma_mix_f32 v23, v171, s85, v23 op_sel:[1,0,0] op_sel_hi:[1,0,0]\n\t"
        "v_fma_mix_f32 v24, v172, s85, v24 op_sel:[0,0,0] op_sel_hi:[1,0,0]\n\t"
        "v_fma_mix_f32 v25, v172, s85, v25 op_sel:[1,0,0] op_sel_hi:[1,0,0]\n\t"
        "v_fma_mix_f32 v26, v173, s85, v26 op_sel:[0,0,0] op_sel_hi:[1,0,0]\n\t"
        "v_fma_mix_f32 v27, v173, s85, v27 op_sel:[1,0,0] op_sel_hi:[1,0,0]\n\t"
        "v_fma_mix_f32 v28, v174, s85, v28 op_sel:[0,0,0] op_sel_hi:[1,0,0]\n\t"
        "v_fma_mix_f32 v29, v174, s85, v29 op_sel:[1,0,0] op_sel_hi:[1,0,0]\n\t"
        "v_fma_mix_f32 v30, v175, s85, v30 op_sel:[0,0,0] op_sel_hi:[1,0,0]\n\t"
        "v_fma_mix_f32 v31, v175, s85, v31 op_sel:[1,0,0] op_sel_hi:[1,0,0]\n\t"
        "Lxf2%=:\n\t"
        "s_bitcmp1_b32 %[da], 0\n\t"
        "s_cbranch_scc0 Lxg0%=\n\t"
        "s_and_b32 %[sw], %[da], 6\n\t"
        "s_cmp_eq_u32 %[sw], 6\n\t"
        "s_cbranch_scc1 Lxh0%=\n\t"
        "s_waitcnt vmcnt(4)\n\t"
        "s_cmp_eq_u32 %[sw], 0\n\t"
        "s_cbranch_scc0 Lxh0%=\n\t"
        "s_waitcnt vmcnt(0)\n\t"
        "Lxh0%=:\n\t"
        "v_fma_mix_f32 v0, v128, s81, v0 op_sel:[0,0,0] op_sel_hi:[1,0,0]\n\t"
        "v_fma_mix_f32 v1, v128, s81, v1 op_sel:[1,0,0] op_sel_hi:[1,0,0]\n\t"
        "v_fma_mix_f32 v2, v129, s81, v2 op_sel:[0,0,0] op_sel_hi:[1,0,0]\n\t"
        "v_fma_mix_f32 v3, v129, s81, v3 op_sel:[1,0,0] op_sel_hi:[1,0,0]\n\t"
        "v_fma_mix_f32 v4, v130, s81, v4 op_sel:[0,0,0] op_sel_hi:[1,0,0]\n\t"
        "v_fma_mix_f32 v5, v130, s81, v5 op_sel:[1,0,0] op_sel_hi:[1,0,0]\n\t"
        "v_fma_mix_f32 v6, v131, s81, v6 op_sel:[0,0,0] op_sel_hi:[1,0,0]\n\t"
        "v_fma_mix_f32 v7, v131, s81, v7 op_sel:[1,0,0] op_sel_hi:[1,0,0]\n\t"
        "v_fma_mix_f32 v8, v132, s81, v8 op_sel:[0,0,0] op_sel_hi:[1,0,0]\n\t"
        "v_fma_mix_f32 v9, v132, s81, v9 op_sel:[1,0,0] op_sel_hi:[1,0,0]\n\t"
        "v_fma_mix_f32 v10, v133, s81, v10 op_sel:[0,0,0] op_sel_hi:[1,0,0]\n\t"
        "v_fma_mix_f32 v11, v133, s81, v11 op_sel:[1,0,0] op_sel_hi:[1,0,0]\n\t"
        "v_fma_mix_f32 v12, v134, s81, v12 op_sel:[0,0,0] op_sel_hi:[1,0,0]\n\t"
        "v_fma_mix_f32 v13, v134, s81, v13 op_sel:[1,0,0] op_sel_hi:[1,0,0]\n\t"
        "v_fma_mix_f32 v14, v135, s81, v14 op_sel:[0,0,0] op_sel_hi:[1,0,0]\n\t"
        "v_fma_mix_f32 v15, v135, s81, v15 op_sel:[1,0,0] op_sel_hi:[1,0,0]\n\t"
        "v_fma_mix_f32 v16, v136, s81, v16 op_sel:[0,0,0] op_sel_hi:[1,0,0]\n\t"
        "v_fma_mix_f32 v17, v136, s81, v17 op_sel:[1,0,0] op_sel_hi:[1,0,0]\n\t"
        "v_fma_mix_f32 v18, v137, s81, v18 op_sel:[0,0,0] op_sel_hi:[1,0,0]\n\t"
        "v_fma_mix_f32 v19, v137, s81, v19 op_sel:[1,0,0] op_sel_hi:[1,0,0]\n\t"
        "v_fma_mix_f32 v20, v138, s81, v20 op_sel:[0,0,0] op_sel_hi:[1,0,0]\n\t"
        "v_fma_mix_f32 v21, v138, s81, v21 op_sel:[1,0,0] op_sel_hi:[1,0,0]\n\t"
        "v_fma_mix_f32 v22, v139, s81, v22 op_sel:[0,0,0] op_sel_hi:[1,0,0]\n\t"
        "v_fma_mix_f32 v23, v139, s81, v23 op_sel:[1,0,0] op_sel_hi:[1,0,0]\n\t"
        "v_fma_mix_f32 v24, v140, s81, v24 op_sel:[0,0,0] op_sel_hi:[1,0,0]\n\t"
        "v_fma_mix_f32 v25, v140, s81, v25 op_sel:[1,0,0] op_sel_hi:[1,0,0]\n\t"
        "v_fma_mix_f32 v26, v141, s81, v26 op_sel:[0,0,0] op_sel_hi:[1,0,0]\n\t"
        "v_fma_mix_f32 v27, v141, s81, v27 op_sel:[1,0,0] op_sel_hi:[1,0,0]\n\t"
        "v_fma_mix_f32 v28, v142, s81, v28 op_sel:[0,0,0] op_sel_hi:[1,0,0]\n\t"
        "v_fma_mix_f32 v29, v142, s81, v29 op_sel:[1,0,0] op_sel_hi:[1,0,0]\n\t"
        "v_fma_mix_f32 v30, v143, s81, v30 op_sel:[0,0,0] op_sel_hi:[1,0,0]\n\t"
        "v_fma_mix_f32 v31, v143, s81, v31 op_sel:[1,0,0] op_sel_hi:[1,0,0]\n\t"
        "Lxg0%=:\n\t"
        "s_bitcmp1_b32 %[da], 1\n\t"
        "s_cbranch_scc0 Lxg1%=\n\t"
        "s_waitcnt vmcnt(4)\n\t"
        "s_bitcmp1_b32 %[da], 2\n\t"
        "s_cbranch_scc1 Lxh1%=\n\t"
        "s_waitcnt vmcnt(0)\n\t"
        "Lxh1%=:\n\t"
        "v_fma_mix_f32 v0, v144, s83, v0 op_sel:[0,0,0] op_sel_hi:[1,0,0]\n\t"
        "v_fma_mix_f32 v1, v144, s83, v1 op_sel:[1,0,0] op_sel_hi:[1,0,0]\n\t"
        "v_fma_mix_f32 v2, v145, s83, v2 op_sel:[0,0,0] op_sel_hi:[1,0,0]\n\t"
        "v_fma_mix_f32 v3, v145, s83, v3 op_sel:[1,0,0] op_sel_hi:[1,0,0]\n\t"
        "v_fma_mix_f32 v4, v146, s83, v4 op_sel:[0,0,0] op_sel_hi:[1,0,0]\n\t"
        "v_fma_mix_f32 v5, v146, s83, v5 op_sel:[1,0,0] op_sel_hi:[1,0,0]\n\t"
        "v_fma_mix_f32 v6, v147, s83, v6 op_sel:[0,0,0] op_sel_hi:[1,0,0]\n\t"
        "v_fma_mix_f32 v7, v147, s83, v7 op_sel:[1,0,0] op_sel_hi:[1,0,0]\n\t"
        "v_fma_mix_f32 v8, v148, s83, v8 op_sel:[0,0,0] op_sel_hi:[1,0,0]\n\t"
        "v_fma_mix_f32 v9, v148, s83, v9 op_sel:[1,0,0] op_sel_hi:[1,0,0]\n\t"
        "v_fma_mix_f32 v10, v149, s83, v10 op_sel:[0,0,0] op_sel_hi:[1,0,0]\n\t"
        "v_fma_mix_f32 v11, v149, s83, v11 op_sel:[1,0,0] op_sel_hi:[1,0,0]\n\t"
        "v_fma_mix_f32 v12, v150, s83, v12 op_sel:[0,0,0] op_sel_hi:[1,0,0]\n\t"
        "v_fma_mix_f32 v13, v150, s83, v13 op_sel:[1,0,0] op_sel_hi:[1,0,0]\n\t"
        "v_fma_mix_f32 v14, v151, s83, v14 op_sel:[0,0,0] op_sel_hi:[1,0,0]\n\t"
        "v_fma_mix_f32 v15, v151, s83, v15 op_sel:[1,0,0] op_sel_hi:[1,0,0]\n\t"
        "v_fma_mix_f32 v16, v152, s83, v16 op_sel:[0,0,0] op_sel_hi:[1,0,0]\n\t"
        "v_fma_mix_f32 v17, v152, s83, v17 op_sel:[1,0,0] op_sel_hi:[1,0,0]\n\t"
        "v_fma_mix_f32 v18, v153, s83, v18 op_sel:[0,0,0] op_sel_hi:[1,0,0]\n\t"
        "v_fma_mix_f32 v19, v153, s83, v19 op_sel:[1,0,0] op_sel_hi:[1,0,0]\n\t"
        "v_fma_mix_f32 v20, v154, s83, v20 op_sel:[0,0,0] op_sel_hi:[1,0,0]\n\t"
        "v_fma_mix_f32 v21, v154, s83, v21 op_sel:[1,0,0] op_sel_hi:[1,0,0]\n\t"
        "v_fma_mix_f32 v22, v155, s83, v22 op_sel:[0,0,0] op_sel_hi:[1,0,0]\n\t"
        "v_fma_mix_f32 v23, v155, s83, v23 op_sel:[1,0,0] op_sel_hi:[1,0,0]\n\t"
        "v_fma_mix_f32 v24, v156, s83, v24 op_sel:[0,0,0] op_sel_hi:[1,0,0]\n\t"
        "v_fma_mix_f32 v25, v156, s83, v25 op_sel:[1,0,0] op_sel_hi:[1,0,0]\n\t"
        "v_fma_mix_f32 v26, v157, s83, v26 op_sel:[0,0,0] op_sel_hi:[1,0,0]\n\t"
        "v_fma_mix_f32 v27, v157, s83, v27 op_sel:[1,0,0] op_sel_hi:[1,0,0]\n\t"
        "v_fma_mix_f32 v28, v158, s83, v28 op_sel:[0,0,0] op_sel_hi:[1,0,0]\n\t"
        "v_fma_mix_f32 v29, v158, s83, v29 op_sel:[1,0,0] op_sel_hi:[1,0,0]\n\t"
        "v_fma_mix_f32 v30, v159, s83, v30 op_sel:[0,0,0] op_sel_hi:[1,0,0]\n\t"
        "v_fma_mix_f32 v31, v159, s83, v31 op_sel:[1,0,0] op_sel_hi:[1,0,0]\n\t"
        "Lxg1%=:\n\t"
        "s_bitcmp1_b32 %[da], 2\n\t"
        "s_cbranch_scc0 Lxe%=\n\t"
        "s_waitcnt vmcnt(3)\n\t"
        "v_fma_mix_f32 v0, v160, s85, v0 op_sel:[0,0,0] op_sel_hi:[1,0,0]\n\t"
        "v_fma_mix_f32 v1, v160, s85, v1 op_sel:[1,0,0] op_sel_hi:[1,0,0]\n\t"
        "v_fma_mix_f32 v2, v161, s85, v2 op_sel:[0,0,0] op_sel_hi:[1,0,0]\n\t"
        "v_fma_mix_f32 v3, v161, s85, v3 op_sel:[1,0,0] op_sel_hi:[1,0,0]\n\t"
        "v_fma_mix_f32 v4, v162, s85, v4 op_sel:[0,0,0] op_sel_hi:[1,0,0]\n\t"
        "v_fma_mix_f32 v5, v162, s85, v5 op_sel:[1,0,0] op_sel_hi:[1,0,0]\n\t"
        "v_fma_mix_f32 v6, v163, s85, v6 op_sel:[0,0,0] op_sel_hi:[1,0,0]\n\t"
        "v_fma_mix_f32 v7, v163, s85, v7 op_sel:[1,0,0] op_sel_hi:[1,0,0]\n\t"
        "s_waitcnt vmcnt(2)\n\t"
        "v_fma_mix_f32 v8, v164, s85, v8 op_sel:[0,0,0] op_sel_hi:[1,0,0]\n\t"
        "v_fma_mix_f32 v9, v164, s85, v9 op_sel:[1,0,0] op_sel_hi:[1,0,0]\n\t"
        "v_fma_mix_f32 v10, v165, s85, v10 op_sel:[0,0,0] op_sel_hi:[1,0,0]\n\t"
        "v_fma_mix_f32 v11, v165, s85, v11 op_sel:[1,0,0] op_sel_hi:[1,0,0]\n\t"
        "v_fma_mix_f32 v12, v166, s85, v12 op_sel:[0,0,0] op_sel_hi:[1,0,0]\n\t"
        "v_fma_mix_f32 v13, v166, s85, v13 op_sel:[1,0,0] op_sel_hi:[1,0,0]\n\t"
        "v_fma_mix_f32 v14, v167, s85, v14 op_sel:[0,0,0] op_sel_hi:[1,0,0]\n\t"
        "v_fma_mix_f32 v15, v167, s85, v15 op_sel:[1,0,0] op_sel_hi:[1,0,0]\n\t"
        "s_waitcnt vmcnt(1)\n\t"
        "v_fma_mix_f32 v16, v168, s85, v16 op_sel:[0,0,0] op_sel_hi:[1,0,0]\n\t"
        "v_fma_mix_f32 v17, v168, s85, v17 op_sel:[1,0,0] op_sel_hi:[1,0,0]\n\t"
        "v_fma_mix_f32 v18, v169, s85, v18 op_sel:[0,0,0] op_sel_hi:[1,0,0]\n\t"
        "v_fma_mix_f32 v19, v169, s85, v19 op_sel:[1,0,0] op_sel_hi:[1,0,0]\n\t"
        "v_fma_mix_f32 v20, v170, s85, v20 op_sel:[0,0,0] op_sel_hi:[1,0,0]\n\t"
        "v_fma_mix_f32 v21, v170, s85, v21 op_sel:[1,0,0] op_sel_hi:[1,0,0]\n\t"
        "v_fma_mix_f32 v22, v171, s85, v22 op_sel:[0,0,0] op_sel_hi:[1,0,0]\n\t"
        "v_fma_mix_f32 v23, v171, s85, v23 op_sel:[1,0,0] op_sel_hi:[1,0,0]\n\t"
        "s_waitcnt vmcnt(0)\n\t"
        "v_fma_mix_f32 v24, v172, s85, v24 op_sel:[0,0,0] op_sel_hi:[1,0,0]\n\t"
        "v_fma_mix_f32 v25, v172, s85, v25 op_sel:[1,0,0] op_sel_hi:[1,0,0]\n\t"
        "v_fma_mix_f32 v26, v173, s85, v26 op_sel:[0,0,0] op_sel_hi:[1,0,0]\n\t"
        "v_fma_mix_f32 v27, v173, s85, v27 op_sel:[1,0,0] op_sel_hi:[1,0,0]\n\t"
        "v_fma_mix_f32 v28, v174, s85, v28 op_sel:[0,0,0] op_sel_hi:[1,0,0]\n\t"
        "v_fma_mix_f32 v29, v174, s85, v29 op_sel:[1,0,0] op_sel_hi:[1,0,0]\n\t"
        "v_fma_mix_f32 v30, v175, s85, v30 op_sel:[0,0,0] op_sel_hi:[1,0,0]\n\t"
        "v_fma_mix_f32 v31, v175, s85, v31 op_sel:[1,0,0] op_sel_hi:[1,0,0]\n\t"
        "Lxe%=:\n\t"
        "v_max3_f32 v178, v0, v1, 0\n\t"
        "v_max3_f32 v178, v178, v2, v3\n\t"
        "v_max3_f32 v178, v178, v4, v5\n\t"
        "v_max3_f32 v178, v178, v6, v7\n\t"
        "v_max3_f32 v178, v178, v8, v9\n\t"
        "v_max3_f32 v178, v178, v10, v11\n\t"
        "v_max3_f32 v178, v178, v12, v13\n\t"
        "v_max3_f32 v178, v178, v14, v15\n\t"
        "v_max3_f32 v178, v178, v16, v17\n\t"
        "v_max3_f32 v178, v178, v18, v19\n\t"
        "v_max3_f32 v178, v178, v20, v21\n\t"
        "v_max3_f32 v178, v178, v22, v23\n\t"
        "v_max3_f32 v178, v178, v24, v25\n\t"
        "v_max3_f32 v178, v178, v26, v27\n\t"
        "v_max3_f32 v178, v178, v28, v29\n\t"
        "v_max3_f32 v178, v178, v30, v31\n\t"
        "s_cmp_lt_u32 %[spin], 0x1000000\n\t"
        "s_cselect_b32 %[sa], 0, -1\n\t"
        "s_waitcnt lgkmcnt(0)\n\t"
        "v_max3_u32 v178, v178, v184, %[sa]\n\t"
        "s_nop 1\n\t"
        "v_max_u32_dpp v178, v178, v178 row_shr:1 row_mask:0xf bank_mask:0xf bound_ctrl:1\n\t"
        "s_nop 1\n\t"
        "v_max_u32_dpp v178, v178, v178 row_shr:2 row_mask:0xf bank_mask:0xf bound_ctrl:1\n\t"
        "s_nop 1\n\t"
        "v_max_u32_dpp v178, v178, v178 row_shr:4 row_mask:0xf bank_mask:0xf bound_ctrl:1\n\t"
        "s_nop 1\n\t"
        "v_max_u32_dpp v178, v178, v178 row_shr:8 row_mask:0xf bank_mask:0xf bound_ctrl:1\n\t"
        "s_nop 1\n\t"
        "v_max_u32_dpp v178, v178, v178 row_bcast:15 row_mask:0xa bank_mask:0xf\n\t"
        "s_nop 1\n\t"
        "v_max_u32_dpp v178, v178, v178 row_bcast:31 row_mask:0xc bank_mask:0xf\n\t"
        "s_mov_b32 exec_lo, 0\n\t"
        "s_mov_b32 exec_hi, 0x80000000\n\t"
        "global_store_dwordx2 v185, v[178:179], %[am1] sc0 sc1\n\t"
        "s_mov_b64 exec, -1\n\t"
        "ds_write_b32 v186, v179 offset:4\n\t"
        "s_waitcnt lgkmcnt(0)"
        : "={v[0:15]}"(ac0), "={v[16:31]}"(ac1), [call] "=&s"(call),
          [step] "=&s"(s_step), [sa] "=&s"(s_sa), [sw] "=&s"(s_sw), [da] "=&s"(s_da), [dw] "=&s"(s_dw), [spin] "=&s"(s_spin)
        : [xa] "{v236}"(xa), [wa] "{v237}"(wa), [fa] "{v238}"(C_FULL + (lane & 1) * 256), [vo0] "{v239}"(lane * 16),
          [fd] "{v240}"((((sq + 1 + lane) & 3) * 16 + lane) * 128),
          [tbl] "s"(IOTA), [m0v] "s"(C_DONE + cw * 256), [cnt] "s"(cnt), [t0] "s"((unsigned)T0),
          [xo1] "s"(xd + ((sq + 1) & 3) * 49152), [xo2] "s"(xd + ((sq + 2) & 3) * 49152 + 4096),
          [xo3] "s"(xd + ((sq + 3) & 3) * 49152 + 8192), [xi] "s"(xd + sq * 49152),
          [xfo] "s"(ws + WS_XF + ((b & ~3) * 4 + cw) * 512), [xfi] "s"(ws + WS_XF + (b * 4 + cw) * 512),
          [am1] "s"(ws + WS_AM1 + b * 128 + cw * 8), [cbad] "n"(C_BAD)
        : "memory", "vcc", "scc", "s80", "s81", "s82", "s83", "s84", "s85", "s86", "s87", "s88", "s89", "s90", "s91", "s92",
          "s93", "s94", "s95",
          "v32", "v33", "v34", "v35", "v36", "v37", "v38", "v39", "v40", "v41", "v42", "v43", "v44", "v45", "v46", "v47",
          "v48", "v49", "v50", "v51", "v52", "v53", "v54", "v55", "v56", "v57", "v58", "v59", "v60", "v61", "v62", "v63",
          "v64", "v65", "v66", "v67", "v68", "v69", "v70", "v71", "v72", "v73", "v74", "v75", "v76", "v77", "v78", "v79",
          "v80", "v81", "v82", "v83", "v84", "v85", "v86", "v87", "v88", "v89", "v90", "v91", "v92", "v93", "v94", "v95",
          "v96", "v97", "v98", "v99", "v100", "v101", "v102", "v103", "v104", "v105", "v106", "v107", "v108", "v109", "v110", "v111",
          "v112", "v113", "v114", "v115", "v116", "v117", "v118", "v119", "v120", "v121", "v122", "v123", "v124", "v125", "v126", "v127",
          "v128", "v129", "v130", "v131", "v132", "v133", "v134", "v135", "v136", "v137", "v138", "v139", "v140", "v141", "v142", "v143",
          "v144", "v145", "v146", "v147", "v148", "v149", "v150", "v151", "v152", "v153", "v154", "v155", "v156", "v157", "v158", "v159",
          "v160", "v161", "v162", "v163", "v164", "v165", "v166", "v167", "v168", "v169", "v170", "v171", "v172", "v173", "v174", "v175",
          "v176", "v177", "v178", "v179", "v180", "v181", "v182", "v183", "v184", "v185", "v186", "v187", "v188", "v189", "v190", "v191",
          "v192", "v193", "v194", "v195", "v196", "v197", "v198", "v199", "v200", "v201", "v202", "v203", "v204", "v205", "v206", "v207",
          "v208", "v209", "v210", "v211", "v212", "v213", "v214", "v215", "v216", "v217", "v218", "v219", "v220", "v221", "v222", "v223",
          "v224", "v225", "v226", "v227", "v228", "v229", "v230", "v231", "v232", "v233", "v234", "v235");
    if (s_spin >= 0x1000000) st_tmo(ws, call);
  }
#pragma unroll
  for (int e = 0; e < 16; ++e) { ac0[e] = __builtin_fmaxf(ac0[e], 0.f); ac1[e] = __builtin_fmaxf(ac1[e], 0.f); }
  char* orow = p.out + (size_t)(mt * 128 + sq * 32 + (lane & 15)) * 4096 + n0 + cw * 64 + g4 * 16;
  asm volatile("" : "+v"(ac0), "+v"(ac1), "+v"(orow));
  unsigned spins = 0, used = POISON;
#pragma nounroll
  while (true) {
    const u32x2 gf = *(volatile LAS u32x2*)(lds + C_GFLAG);
    unsigned st = uni(gf.x);
    const unsigned f = uni(gf.y);
    if (++spins == 0x2000000) st = 2;
    if (st == 2) {
      const u32x4 o = u32x4{0x80808080u, 0x80808080u, 0x80808080u, 0x80808080u};
      asm volatile("s_waitcnt vmcnt(0)" ::: "memory");
      OUT_STORE(orow, o, "nt");
      OUT_STORE(orow + 16 * 4096, o, "nt");
      break;
    }
    if (st != 0 && f != used) {
      used = f;
      const float inv = __uint_as_float(f);
#pragma unroll
      for (int i = 0; i < 2; ++i) {
        unsigned P[4];
#pragma unroll
        for (int j = 0; j < 4; ++j) {
          unsigned q[4];
#pragma unroll
          for (int e = 0; e < 4; ++e)
            q[e] = __float_as_uint(__builtin_fmaf(i ? ac1[4 * j + e] : ac0[4 * j + e], inv, 12582912.0f));
          P[j] = __builtin_amdgcn_perm(q[1], q[0], 0x0c0c0400u) | __builtin_amdgcn_perm(q[3], q[2], 0x04000c0cu);
        }
        auto s0 = __builtin_amdgcn_permlane32_swap(P[0], P[2], false, false);
        auto s1 = __builtin_amdgcn_permlane32_swap(P[1], P[3], false, false);
        auto q0 = __builtin_amdgcn_permlane16_swap(s0[0], s1[0], false, false);
        auto q1 = __builtin_amdgcn_permlane16_swap(s0[1], s1[1], false, false);
        asm volatile("s_waitcnt vmcnt(1)" ::: "memory");
        const u32x4 o = u32x4{q0[0], q0[1], q1[0], q1[1]};
        if (st == 4) OUT_STORE(orow + i * 16 * 4096, o, "sc0 sc1");
        else OUT_STORE(orow + i * 16 * 4096, o, "nt");
      }
    }
    if (st == 1) break;
  }
}

extern "C" void kernel_launch(int rank, int nranks, ncclComm_t comm, hipStream_t stream, void* const* d_in,
                              const int* in_sizes, int n_in, void* d_out, int out_size, const int* in_dtypes,
                              int out_dtype, void* const* d_out_all, void* const* d_ws_all, size_t ws_bytes) {
  if (nranks != NR || (unsigned)rank >= (unsigned)NR || ws_bytes < WS_END || n_in != 2 || in_sizes[0] != 4096 * 512 ||
      in_sizes[1] != 4096 * 4096 || out_size != 512 * 4096)
    return;
  static bool init = false;
  if (!init) {
    HIP_CHECK(hipFuncSetAttribute((const void*)a2a_gemm, hipFuncAttributeMaxDynamicSharedMemorySize, LDS_BYTES));
    init = true;
  }
  Params p;
  p.x = (const char*)d_in[0];
  p.w = (const char*)d_in[1];
  p.out = (char*)d_out;
  p.dbg = nullptr;
  p.rank = rank;
  p.pad0 = p.pad1 = p.pad2 = 0;
  p.pad3 = p.pad4 = 0;
  for (int i = 0; i < NR; ++i) p.wsrel[i] = (char*)d_ws_all[(rank - i) & 7];
  make_sched(p.sched);
  hipLaunchKernelGGL(a2a_gemm, dim3(NWG), dim3(NTHR), LDS_BYTES, stream, p);
  HIP_CHECK(hipGetLastError());
}
```
